```python
import numpy as np

kernel_name = "gpu_gemm_a2a_m4096_k4096_n16384_e4m3e4m3_dequant"


ABSMAX_RTOL = 0.03

COMPUTE_DTYPE = "float32"
EPILOGUE = "dequant"
IN_QUANT = None
OUT_QUANT = None
IN_FP8 = "e4m3e4m3"
IN_DTYPES = {"x": "e4m3", "w_mat": "e4m3",
             "scale_x": "float32", "scale_w": "float32"}
WORLD_SIZE = 8

def _snap_fp8(x, mant_bits, max_abs, min_normal):
    a = np.abs(x.astype(np.float64))
    _m, _e = np.frexp(a)
    q = np.where(a >= min_normal, np.ldexp(1.0, _e - 1 - mant_bits),
                 min_normal * 2.0 ** -mant_bits)
    snapped = np.minimum(np.round(a / q) * q, max_abs)
    return (np.sign(x) * snapped).astype(np.float32)

def setup_inputs():

    x = _snap_fp8(np.random.randn(4096, 4096).astype(np.float32), 3, 448.0, 2.0 ** -6)
    w_mat = _snap_fp8(np.random.randn(4096, 16384).astype(np.float32), 3, 448.0, 2.0 ** -6)
    scale_x = (0.02 * (1.0 + np.random.rand(1))).astype(np.float32)
    scale_w = (0.02 * (1.0 + np.random.rand(1))).astype(np.float32)
    return {"x": x, "w_mat": w_mat, "scale_x": scale_x, "scale_w": scale_w}

def reference(x, w_mat, scale_x, scale_w):
    acc = (x @ w_mat).astype(np.float32)
    y = (acc * (scale_x * scale_w)).astype(np.float32)
    return y
```

```cpp
#include <hip/hip_runtime.h>
#include <rccl/rccl.h>
#include <cstdio>

#define HIP_CHECK(expr) do { hipError_t _e = (expr); if (_e != hipSuccess) { \
    fprintf(stderr, "HIP error %s at %s:%d\n", hipGetErrorString(_e), __FILE__, __LINE__); \
    return; } } while (0)

#define MAP_CMAJOR 1
#define OUT_AUX 16
#define A_AUX 17
#define SEND_L 1

namespace {

typedef unsigned u32x4 __attribute__((ext_vector_type(4)));
typedef int v8i __attribute__((ext_vector_type(8)));
typedef int v4i __attribute__((ext_vector_type(4)));
typedef int v2i __attribute__((ext_vector_type(2)));
typedef float f32x4 __attribute__((ext_vector_type(4)));
#define LAS __attribute__((address_space(3)))
#define DI __device__ __forceinline__

constexpr int NR = 8;
constexpr int KDIM = 4096, NGLOB = 16384, NLOC = 2048, MLOC = 512;
constexpr unsigned SHARD = MLOC * KDIM;
constexpr unsigned BLK = 16 * KDIM;
constexpr int NBLK = MLOC / 16;
constexpr int NSEND = 32, NWG = 256;
constexpr size_t FLAG_OFF = 0;
constexpr size_t TMO_OFF = 16 << 10;
constexpr size_t CTR_OFF = 64 << 10;
constexpr size_t DBG_OFF = 512 << 10;
constexpr size_t DATA_OFF = 1 << 20;
constexpr size_t SLOT_BYTES = (size_t)NR * SHARD;
constexpr size_t WS_NEED = DATA_OFF + 2 * SLOT_BYTES;
constexpr unsigned long long DEADLINE_TICKS = 50000000ull;
constexpr int STAGE_PER_WAVE = 32768;
constexpr int RED_OFF = 4 * STAGE_PER_WAVE;
constexpr int LDS_BYTES = RED_OFF + 2 * 4 * 4096;

struct Params {
  const unsigned char* x;
  const unsigned char* w;
  const float* sx;
  const float* sw;
  float* out;
  unsigned char* myws;
  unsigned char* ws[NR];
  int rank;
  int pad;
};

template <int V> struct IC { static constexpr int value = V; };
template <int I, int N, class F> DI void static_for(F&& f) {
  if constexpr (I < N) { f(IC<I>{}); static_for<I + 1, N>(f); }
}
template <int N> DI void waitv() { asm volatile("s_waitcnt vmcnt(%0)" :: "i"(N) : "memory"); }
DI void pin(u32x4& v) { asm volatile("" : "+v"(v)); }
DI void ld16(u32x4& v, unsigned voff, const void* sbase) {
  asm volatile("global_load_dwordx4 %0, %1, %2" : "=v"(v) : "v"(voff), "s"(sbase) : "memory");
}
DI void st16_sys(unsigned voff, void* sbase, u32x4 v) {
  asm volatile("global_store_dwordx4 %0, %1, %2 sc0 sc1\n\ts_nop 1" :: "v"(voff), "v"(v), "s"(sbase) : "memory");
}
DI void st_flag(unsigned voff, void* sbase, unsigned v) {
  asm volatile("global_store_dword %0, %1, %2 sc0 sc1" :: "v"(voff), "v"(v), "s"(sbase) : "memory");
}
DI void glds16(unsigned voff, const void* sbase, unsigned lds_dst) {
  unsigned keep;
  asm volatile("s_mov_b32 %0, m0\n\ts_mov_b32 m0, %3\n\ts_nop 0\n\tglobal_load_lds_dwordx4 %1, %2\n\ts_mov_b32 m0, %0"
               : "=&s"(keep) : "v"(voff), "s"(sbase), "s"(lds_dst) : "memory");
}
DI unsigned min4(unsigned v) {
  unsigned a = __builtin_amdgcn_readlane(v, 0), b = __builtin_amdgcn_readlane(v, 1);
  unsigned c = __builtin_amdgcn_readlane(v, 2), d = __builtin_amdgcn_readlane(v, 3);
  a = a < b ? a : b; c = c < d ? c : d;
  return __builtin_amdgcn_readfirstlane(a < c ? a : c);
}
DI unsigned poll(const unsigned* p) { return __hip_atomic_load(p, __ATOMIC_RELAXED, __HIP_MEMORY_SCOPE_SYSTEM); }

DI unsigned spin(const unsigned* fl, unsigned need, unsigned* tmo, unsigned call, unsigned& poison) {
  const unsigned long long t0 = __builtin_amdgcn_s_memrealtime();
  unsigned it = 0;
  for (;;) {
    unsigned a = min4(poll(fl));
    if (a >= need) return a;
    if ((++it & 63u) == 0u) {
      unsigned t = __builtin_amdgcn_readfirstlane(__hip_atomic_load(tmo, __ATOMIC_RELAXED, __HIP_MEMORY_SCOPE_AGENT));
      if (t == call || __builtin_amdgcn_s_memrealtime() - t0 > DEADLINE_TICKS) {
        __hip_atomic_store(tmo, call, __ATOMIC_RELAXED, __HIP_MEMORY_SCOPE_AGENT);
        poison = 0x7fc00000u;
        return 0xffffffffu;
      }
    }
    __builtin_amdgcn_s_sleep(1);
  }
}

#define TS(idx) do { } while (0)

struct Ctx {
  __amdgpu_buffer_rsrc_t rsA, rsX, rsO;
  const unsigned* flp;
  unsigned* tmo;
  LAS unsigned char* red;
  unsigned call, fbase, avail, pollv;
  unsigned voffA, voffX, voffO, wr_off, rd_off;
  int n, nloc, rowL, rowR, row_own, src;
  unsigned poison;
  float scale;
};

DI unsigned blk_row0(const Ctx& c, int i) {
  const bool loc = i < c.nloc;
  const int base = loc ? c.rowL : c.rowR, mul = loc ? 112 : 16;
  return (unsigned)(base + mul * i);
}

struct Src { __amdgpu_buffer_rsrc_t rs; unsigned voff, soff, step; };
DI Src src_of(const Ctx& c, int j) {
  const bool loc = j < c.nloc;
  Src s;
  s.rs = loc ? c.rsX : c.rsA;
  s.voff = loc ? c.voffX : c.voffA;
  s.soff = (blk_row0(c, j) - (loc ? (unsigned)c.row_own : 0u)) * KDIM;
  s.step = loc ? 64u : 1024u;
  return s;
}
DI void load_frag(const Src& s, v8i& A, int kb) {
  v4i lo = __builtin_bit_cast(v4i, __builtin_amdgcn_raw_buffer_load_b128(s.rs, s.voff, s.soff + (2 * kb) * s.step, A_AUX));
  v4i hi = __builtin_bit_cast(v4i, __builtin_amdgcn_raw_buffer_load_b128(s.rs, s.voff, s.soff + (2 * kb + 1) * s.step, A_AUX));
  A = __builtin_shufflevector(lo, hi, 0, 1, 2, 3, 4, 5, 6, 7);
}

DI void ensure(Ctx& c, int j) {
  const unsigned need = (j < c.nloc) ? 0u : c.fbase + (unsigned)(j - c.nloc) + 1u;
  const unsigned seen = min4(c.pollv);
  c.avail = c.avail > seen ? c.avail : seen;
  if (c.avail < need) c.avail = spin(c.flp, need, c.tmo, c.call, c.poison);
  asm volatile("" ::: "memory");
}

DI void step(Ctx& c, const v8i (&B)[32], int i, const v8i (&Ac)[8], v8i (&An)[8]) {
  const int j = (i + 2 < c.n) ? (i + 2) : (c.n - 1);
  ensure(c, j);
  c.pollv = poll(c.flp);
  const Src sj = src_of(c, j);
  f32x4 acc[4];
  const float pz = __builtin_bit_cast(float, __builtin_amdgcn_readfirstlane(c.poison));
  const f32x4 c0 = {pz, pz, pz, pz};
  __builtin_amdgcn_sched_barrier(0);
#pragma unroll
  for (int kb = 0; kb < 8; ++kb) {
    v8i a = Ac[kb];
    load_frag(sj, An[kb], kb);
#pragma unroll
    for (int cb = 0; cb < 4; ++cb)
      acc[cb] = __builtin_amdgcn_mfma_scale_f32_16x16x128_f8f6f4(B[kb * 4 + cb], a, kb == 0 ? c0 : acc[cb], 0, 0, 0, 0, 0, 0);
    __builtin_amdgcn_sched_barrier(0);
  }
  LAS unsigned char* red = c.red + (i & 1) * 16384;
#pragma unroll
  for (int cb = 0; cb < 4; ++cb) *(LAS f32x4*)(red + (c.wr_off ^ (cb * 64))) = acc[cb];
  asm volatile("s_waitcnt lgkmcnt(0)" ::: "memory");
  __builtin_amdgcn_s_barrier();
  asm volatile("" ::: "memory");
  f32x4 s0 = *(LAS f32x4*)(red + c.rd_off), s1 = *(LAS f32x4*)(red + c.rd_off + 4096);
  f32x4 s2 = *(LAS f32x4*)(red + c.rd_off + 8192), s3 = *(LAS f32x4*)(red + c.rd_off + 12288);
  f32x4 r = ((s0 + s1) + (s2 + s3)) * c.scale;
  __builtin_amdgcn_raw_buffer_store_b128(__builtin_bit_cast(u32x4, r), c.rsO, c.voffO, blk_row0(c, i) * (NLOC * 4), OUT_AUX);
  __builtin_amdgcn_sched_barrier(0);
}

__global__ __launch_bounds__(256, 1) void gemm_ag_kernel(Params p) {
  __shared__ __attribute__((aligned(16))) unsigned char smem[LDS_BYTES];
  const int lane = threadIdx.x & 63;
  const int wid = __builtin_amdgcn_readfirstlane(threadIdx.x >> 6);
  const int bid = blockIdx.x;
  const unsigned rank = (unsigned)p.rank;
  if (rank >= (unsigned)NR) return;
  unsigned char* myws = p.myws;
  TS(0);
  const unsigned tick = __hip_atomic_fetch_add((unsigned*)(myws + CTR_OFF) + bid * 256 + threadIdx.x, 1u,
                                               __ATOMIC_RELAXED, __HIP_MEMORY_SCOPE_AGENT);

  if (bid < NSEND) {
    constexpr int S = 4, L = SEND_L;
    if ((bid & 7) == 0) return;
    const unsigned d = (rank + (unsigned)(bid & 7)) & 7u;
    const int slice = (bid >> 3) * 4 + wid;
    const unsigned char* src = p.x + slice * 256;
    const unsigned svoff = (lane & 15) * KDIM + (lane >> 4) * 16;
    u32x4 r[L + 1][S];
    static_for<0, L>([&](auto B_) { constexpr int b = B_.value;
      static_for<0, S>([&](auto S_) { constexpr int s = S_.value; ld16(r[b][s], svoff, src + (size_t)b * BLK + s * 64); }); });
    const unsigned call = __builtin_amdgcn_readfirstlane(tick) + 1u;
    unsigned char* dws = p.ws[d];
    unsigned char* dst = dws + DATA_OFF + (size_t)(call & 1u) * SLOT_BYTES + (size_t)rank * SHARD + slice * 4096;
    unsigned char* flag = dws + FLAG_OFF + (rank * 16 + slice) * 128;
    const unsigned dvoff = lane * 16;
    const unsigned fbase = (call - 1u) * NBLK;
    const unsigned zero = 0;
    static_for<0, NBLK>([&](auto B_) { constexpr int b = B_.value;
      if constexpr (b + L < NBLK)
        static_for<0, S>([&](auto S_) { constexpr int s = S_.value;
          ld16(r[(b + L) % (L + 1)][s], svoff, src + (size_t)(b + L) * BLK + s * 64); });
      constexpr int nl = ((b + L < NBLK) ? (b + L) : (NBLK - 1)) - b;
      constexpr int ns = b - ((b - L > 0) ? (b - L) : 0);
      waitv<(nl + ns) * S>();
      static_for<0, S>([&](auto S_) { constexpr int s = S_.value; pin(r[b % (L + 1)][s]); });
      if constexpr (b - L - 1 >= 0) { if (lane == 0) st_flag(zero, flag, fbase + (b - L - 1) + 1); }
      static_for<0, S>([&](auto S_) { constexpr int s = S_.value;
        st16_sys(dvoff, dst + (size_t)b * BLK + s * 1024, r[b % (L + 1)][s]); });
    });
    static_for<0, L + 1>([&](auto J_) { constexpr int j = J_.value;
      waitv<(L - j) * S>();
      if (lane == 0) st_flag(zero, flag, fbase + (NBLK - 1 - L + j) + 1);
    });
    waitv<0>();
    TS(1);
    return;
  }

  const int xcd = bid & 7, jx = (bid - NSEND) >> 3;
  const int cg = xcd * 4 + (jx & 3), g = jx >> 2;
  LAS unsigned char* lds = (LAS unsigned char*)smem;
  const int q = wid;
  v8i B[32];
  {
    const unsigned char* wb = p.w + (size_t)q * 1024 * NGLOB + rank * NLOC + cg * 64;
    const unsigned sw0 = (unsigned)((lane & 3) ^ ((lane >> 4) & 1));
    const unsigned voff0 = (lane >> 2) * NGLOB + sw0 * 16, voff1 = (lane >> 2) * NGLOB + (sw0 ^ 2u) * 16;
    const unsigned stage = (unsigned)(size_t)lds + q * STAGE_PER_WAVE;
    auto dma = [&](int ch) {
#pragma unroll
      for (int i = 0; i < 16; ++i)
        glds16((i & 1) ? voff1 : voff0, wb + (size_t)(ch * 256 + i * 16) * NGLOB, stage + (ch & 1) * 16384 + i * 1024);
    };
    const int g4 = lane >> 4, i16 = lane & 15;
    const int f = ((i16 >> 3) & 1) | ((g4 & 1) << 1);
    LAS unsigned char* rbase = lds + q * STAGE_PER_WAVE + (16 * g4 + (i16 >> 1)) * 64 + (i16 & 1) * 8;
    auto rd = [&](auto CH) {
      constexpr int ch = CH.value;
#pragma unroll
      for (int kk = 0; kk < 2; ++kk) {
        v8i t[4];
#pragma unroll
        for (int cb = 0; cb < 4; ++cb) {
          LAS unsigned char* a = rbase + (ch & 1) * 16384 + kk * 8192 + ((cb ^ f) * 16);
          v2i r0 = __builtin_amdgcn_ds_read_tr8_b64_v2i32((LAS v2i*)(a));
          v2i r1 = __builtin_amdgcn_ds_read_tr8_b64_v2i32((LAS v2i*)(a + 8 * 64));
          v2i r2 = __builtin_amdgcn_ds_read_tr8_b64_v2i32((LAS v2i*)(a + 64 * 64));
          v2i r3 = __builtin_amdgcn_ds_read_tr8_b64_v2i32((LAS v2i*)(a + 72 * 64));
          t[cb] = (v8i){r0.x, r0.y, r1.x, r1.y, r2.x, r2.y, r3.x, r3.y};
        }
        v8i o0, o1, o2, o3;
        if (ch == 3 && kk == 1)
          asm volatile("" : "=a"(o0), "=a"(o1), "=v"(o2), "=v"(o3) : "0"(t[0]), "1"(t[1]), "2"(t[2]), "3"(t[3]) : "memory");
        else
          asm volatile("" : "=a"(o0), "=a"(o1), "=a"(o2), "=a"(o3) : "0"(t[0]), "1"(t[1]), "2"(t[2]), "3"(t[3]) : "memory");
        B[(ch * 2 + kk) * 4 + 0] = o0; B[(ch * 2 + kk) * 4 + 1] = o1; B[(ch * 2 + kk) * 4 + 2] = o2; B[(ch * 2 + kk) * 4 + 3] = o3;
      }
    };
    dma(0); dma(1);
    waitv<16>(); rd(IC<0>{}); dma(2);
    waitv<16>(); rd(IC<1>{}); dma(3);
    waitv<16>(); rd(IC<2>{});
    waitv<0>();  rd(IC<3>{});
  }
  TS(1);

  Ctx c;
  c.call = __builtin_amdgcn_readfirstlane(tick) + 1u;
  c.fbase = (c.call - 1u) * NBLK;
  c.src = (int)((rank + 1u + (unsigned)g) & 7u);
  c.nloc = (g < 4) ? 5 : 4; c.n = c.nloc + NBLK;
  c.row_own = (int)rank * MLOC;
  c.rowL = (int)rank * MLOC + g * 16; c.rowR = c.src * MLOC - c.nloc * 16;
  c.tmo = (unsigned*)(myws + TMO_OFF);
  c.rsA = __builtin_amdgcn_make_buffer_rsrc(myws + DATA_OFF + (size_t)(c.call & 1u) * SLOT_BYTES, 0, (unsigned)SLOT_BYTES, 0x00020000);
  c.rsO = __builtin_amdgcn_make_buffer_rsrc(p.out, 0, (unsigned)(NR * MLOC * NLOC * 4), 0x00020000);
  c.rsX = __builtin_amdgcn_make_buffer_rsrc((void*)p.x, 0, SHARD, 0x00020000);
  c.voffA = q * 16384 + lane * 16;
  c.voffX = q * 1024 + (lane & 15) * KDIM + (lane >> 4) * 16;
  c.voffO = (4 * q + (lane >> 4)) * (NLOC * 4) + cg * 256 + (lane & 15) * 16;
  c.red = lds + RED_OFF;
  {
    const int m = lane & 15, g4 = lane >> 4;
    c.wr_off = q * 4096 + m * 256 + ((g4 ^ (m & 7)) * 16);
    const int r = 4 * q + g4;
    c.rd_off = r * 256 + ((m ^ (r & 7)) * 16);
  }
  c.scale = p.sx[0] * p.sw[0];
  c.poison = 0u;
  c.flp = (const unsigned*)(myws + FLAG_OFF + (c.src * 16 + q * 4 + (lane & 3)) * 128);
  c.avail = 0u;
  c.pollv = poll(c.flp);
  TS(2);

  v8i A0[8], A1[8], A2[8];
#pragma unroll
  for (int kb = 0; kb < 8; ++kb) load_frag(src_of(c, 0), A0[kb], kb);
#pragma unroll
  for (int kb = 0; kb < 8; ++kb) load_frag(src_of(c, 1), A1[kb], kb);
  for (int i = 0; i < 36; i += 3) {
    step(c, B, i, A0, A2); TS(3 + i);
    step(c, B, i + 1, A1, A0); TS(4 + i);
    step(c, B, i + 2, A2, A1); TS(5 + i);
  }
  if (c.n == 37) { step(c, B, 36, A0, A2); TS(39); }
}

}

extern "C" void kernel_launch(int rank, int nranks, ncclComm_t comm,
                              hipStream_t stream,
                              void* const* d_in, const int* in_sizes, int n_in,
                              void* d_out, int out_size,
                              const int* in_dtypes, int out_dtype,
                              void* const* d_out_all,
                              void* const* d_ws_all, size_t ws_bytes) {
  (void)comm; (void)in_dtypes; (void)out_dtype; (void)d_out_all;
  if (nranks != NR || (unsigned)rank >= (unsigned)NR || n_in < 4 || ws_bytes < WS_NEED ||
      in_sizes[0] != (int)SHARD || in_sizes[1] != KDIM * NGLOB || out_size != NR * MLOC * NLOC) {
    fprintf(stderr, "kernel_launch: unexpected problem shape\n");
    return;
  }
  Params p;
  p.x = (const unsigned char*)d_in[0];
  p.w = (const unsigned char*)d_in[1];
  p.sx = (const float*)d_in[2];
  p.sw = (const float*)d_in[3];
  p.out = (float*)d_out;
  for (int r = 0; r < NR; ++r) p.ws[r] = (unsigned char*)d_ws_all[r];
  p.myws = p.ws[rank];
  p.rank = rank;
  p.pad = 0;
  hipLaunchKernelGGL(gemm_ag_kernel, dim3(NWG), dim3(256), 0, stream, p);
  HIP_CHECK(hipGetLastError());
}
```

```cpp
#include <hip/hip_runtime.h>
#include <rccl/rccl.h>
#include <cstdio>

#define HIP_CHECK(expr) do { hipError_t _e = (expr); if (_e != hipSuccess) { \
    fprintf(stderr, "HIP error %s at %s:%d\n", hipGetErrorString(_e), __FILE__, __LINE__); \
    return; } } while (0)

#define OUT_AUX 16
#define A_AUX 17
#define PREFETCH_W 1
#define SEND_L 1
#define OWNMASK 0x920
#define OWNMASK2 0x120
#define XCOLS 4

namespace {

typedef unsigned u32x4 __attribute__((ext_vector_type(4)));
typedef int v8i __attribute__((ext_vector_type(8)));
typedef int v4i __attribute__((ext_vector_type(4)));
typedef int v2i __attribute__((ext_vector_type(2)));
typedef float f32x4 __attribute__((ext_vector_type(4)));
#define LAS __attribute__((address_space(3)))
#define GAS __attribute__((address_space(1)))
#define DI __device__ __forceinline__
#define SB() __builtin_amdgcn_sched_barrier(0)

constexpr int NR = 8;
constexpr int KDIM = 4096, NGLOB = 16384, NLOC = 2048, MLOC = 512;
constexpr unsigned SHARD = MLOC * KDIM;
constexpr unsigned BLK = 16 * KDIM;
constexpr int NBLK = MLOC / 16;
constexpr int NSEND = 32, NWG = 256;
constexpr unsigned OUT_BYTES = NR * MLOC * NLOC * 4;
constexpr size_t FLAG_OFF = 0;
constexpr size_t TMO_OFF = 16 << 10;
constexpr size_t CTR_OFF = 64 << 10;
constexpr size_t DBG_OFF = 512 << 10;
constexpr size_t DATA_OFF = 1 << 20;
constexpr size_t SLOT_BYTES = (size_t)NR * SHARD;
constexpr size_t WS_NEED = DATA_OFF + 2 * SLOT_BYTES;
constexpr unsigned long long DEADLINE_TICKS = 50000000ull;
constexpr int STAGE_PER_WAVE = 32768;
constexpr int RED_OFF = 4 * STAGE_PER_WAVE;
constexpr int LDS_BYTES = RED_OFF + 4 * 8192;

struct Params {
  const unsigned char* x;
  const unsigned char* w;
  const float* sx;
  const float* sw;
  float* out;
  unsigned char* myws;
  unsigned char* ws[NR];
  int rank;
  int pad;
};

template <int V> struct IC { static constexpr int value = V; };
template <int I, int N, class F> DI void static_for(F&& f) {
  if constexpr (I < N) { f(IC<I>{}); static_for<I + 1, N>(f); }
}
template <int N> DI void waitv() { asm volatile("s_waitcnt vmcnt(%0)" :: "i"(N) : "memory"); }
DI void pin(u32x4& v) { asm volatile("" : "+v"(v)); }
DI void ld16(u32x4& v, unsigned voff, const void* sbase) {
  asm volatile("global_load_dwordx4 %0, %1, %2" : "=v"(v) : "v"(voff), "s"(sbase) : "memory");
}
DI void st16_sys(unsigned voff, void* sbase, u32x4 v) {
  asm volatile("global_store_dwordx4 %0, %1, %2 sc0 sc1\n\ts_nop 1" :: "v"(voff), "v"(v), "s"(sbase) : "memory");
}
DI void st_flag(unsigned voff, void* sbase, unsigned v) {
  asm volatile("global_store_dword %0, %1, %2 sc0 sc1" :: "v"(voff), "v"(v), "s"(sbase) : "memory");
}
DI void glds16(unsigned voff, const void* sbase, unsigned lds_dst) {
  unsigned keep;
  asm volatile("s_mov_b32 %0, m0\n\ts_mov_b32 m0, %3\n\ts_nop 0\n\tglobal_load_lds_dwordx4 %1, %2\n\ts_mov_b32 m0, %0"
               : "=&s"(keep) : "v"(voff), "s"(sbase), "s"(lds_dst) : "memory");
}
DI void lanes4(unsigned v, unsigned (&r)[4]) {
  asm volatile("v_readlane_b32 %0, %4, 0\n\tv_readlane_b32 %1, %4, 1\n\tv_readlane_b32 %2, %4, 2\n\tv_readlane_b32 %3, %4, 3"
               : "=&s"(r[0]), "=&s"(r[1]), "=&s"(r[2]), "=&s"(r[3]) : "v"(v));
}
DI unsigned smin4(const unsigned (&r)[4]) {
  unsigned t, u;
  asm volatile("s_min_u32 %0, %2, %3\n\ts_min_u32 %1, %4, %5\n\ts_min_u32 %0, %0, %1"
               : "=&s"(t), "=&s"(u) : "s"(r[0]), "s"(r[1]), "s"(r[2]), "s"(r[3]) : "scc");
  return t;
}

#define TS(idx) do { } while (0)
#define TS_OUT() do { } while (0)

template <int L>
DI void send_stream(const unsigned char* src, unsigned char* dst0, unsigned char* flag, unsigned tick, int lane) {
  constexpr int S = 4, R = L + 1;
  static_assert(NBLK % R == 0 && 2 * L * S <= 63, "ring");
  const unsigned svoff = (lane & 15) * KDIM + (lane >> 4) * 16, dvoff = lane * 16, zero = 0;
  u32x4 r[R][S];
  auto loads = [&](auto R_, int b) { static_for<0, S>([&](auto S_) {
    ld16(r[R_.value][S_.value], svoff, src + (size_t)b * BLK + S_.value * 64); }); };
  static_for<0, L>([&](auto B_) { loads(B_, B_.value); });
  asm volatile("" : "+v"(tick));
  const unsigned call = __builtin_amdgcn_readfirstlane(tick) + 1u;
  unsigned char* dst = dst0 + (size_t)(call & 1u) * SLOT_BYTES;
  const unsigned fbase = (call - 1u) * NBLK;
  auto stores = [&](auto R_, int b) {
    static_for<0, S>([&](auto S_) { st16_sys(dvoff, dst + (size_t)b * BLK + S_.value * 1024, r[R_.value][S_.value]); }); };
  auto pins = [&](auto R_) { static_for<0, S>([&](auto S_) { pin(r[R_.value][S_.value]); }); };
  static_for<0, R>([&](auto B_) { constexpr int b = B_.value;
    loads(IC<(b + L) % R>{}, b + L);
    waitv<(L + b < 2 * L ? L + b : 2 * L) * S>();
    pins(B_); stores(B_, b); });
  for (int b0 = R; b0 < NBLK - R; b0 += R)
    static_for<0, R>([&](auto B_) { constexpr int k = B_.value; const int b = b0 + k;
      loads(IC<(k + L) % R>{}, b + L);
      waitv<2 * L * S>();
      pins(B_);
      if (lane == 0) st_flag(zero, flag, fbase + (unsigned)(b - L));
      stores(B_, b); });
  static_for<0, R>([&](auto B_) { constexpr int k = B_.value, b = NBLK - R + k;
    if constexpr (b + L < NBLK) loads(IC<(k + L) % R>{}, b + L);
    waitv<((b + L < NBLK ? L : NBLK - 1 - b) + L) * S>();
    pins(B_);
    if (lane == 0) st_flag(zero, flag, fbase + (unsigned)(b - L));
    stores(B_, b); });
  static_for<0, R>([&](auto J_) { constexpr int j = J_.value;
    waitv<(L - j) * S>();
    if (lane == 0) st_flag(zero, flag, fbase + (unsigned)(NBLK - L + j)); });
  waitv<0>();
}

constexpr int LSLOT[16] = {27, 29, 32, 34, 37, 39, 42, 44, 47, 49, 52, 54, 57, 59, 61, 63};
struct Sched {
  int li[64], kb[64], cb[64], last_kb[8], wr[8];
  constexpr Sched() : li(), kb(), cb(), last_kb(), wr() {
    int l = 0, r = 0;
    for (int t = 0; t < 64; ++t) {
      if (l < 16 && LSLOT[l] == t) { li[t] = l; kb[t] = l < 8 ? 4 : 7; cb[t] = l % 8; ++l; }
      else { li[t] = -1; kb[t] = r / 8 + (r >= 32 ? 1 : 0); cb[t] = r % 8; ++r; }
      last_kb[kb[t]] = t;
      wr[cb[t]] = t + 3;
    }
  }
};
constexpr Sched SCH;
constexpr int LDGAP[16] = {10, 12, 18, 20, 24, 26, 36, 38, 45, 46, 48, 50, 0, 2, 4, 6};
constexpr bool sched_ok() {
  for (int p = 0; p < 12; ++p) if (LDGAP[p] < SCH.last_kb[p / 2]) return false;
  return SCH.wr[5] < 64 && SCH.wr[6] == 64 && SCH.wr[7] == 66;
}
static_assert(sched_ok(), "schedule");

struct Item { unsigned off, need, own; };
struct Ctx {
  unsigned long long baseA;
  unsigned long long baseX;
  unsigned sizeA, sizeX;
  __amdgpu_buffer_rsrc_t rsO, rsF;
  GAS unsigned* tmo;
  unsigned call, dead;
  unsigned voffA, voffX, voffO, voffF, wr0, wr1, rd;
  unsigned ownmask, n;
  unsigned rem_off, rem_need, own_off;
  float scale;
};

DI unsigned poll_flags(const Ctx& c) {
  asm volatile("" ::: "memory");
  const unsigned v = __builtin_amdgcn_raw_buffer_load_b32(c.rsF, c.voffF, 0, 17);
  asm volatile("" ::: "memory");
  return v;
}
DI bool spin(const Ctx& c, const Item& it) {
  const unsigned long long t0 = __builtin_amdgcn_s_memrealtime();
  for (unsigned k = 1;; ++k) {
    unsigned fl[4];
    lanes4(poll_flags(c), fl);
    if (smin4(fl) >= it.need) return false;
    if ((k & 63u) == 0u) {
      GAS unsigned* tmo = c.tmo; unsigned call = c.call;
      asm volatile("" : "+s"(tmo), "+s"(call));
      const unsigned t = __builtin_amdgcn_readfirstlane(__hip_atomic_load(tmo, __ATOMIC_RELAXED, __HIP_MEMORY_SCOPE_AGENT));
      const unsigned long long dt = __builtin_amdgcn_s_memrealtime() - t0;
      if (t == call || (unsigned)(dt >> 32) != 0u || (unsigned)dt > (unsigned)DEADLINE_TICKS) {
        __hip_atomic_store(tmo, call, __ATOMIC_RELAXED, __HIP_MEMORY_SCOPE_AGENT);
        return true;
      }
    }
    __builtin_amdgcn_s_sleep(1);
  }
}
DI void item_a(const Ctx& c, unsigned j, Item& it) {
  it.own = 0u - ((c.ownmask >> j) & 1u);
  it.off = (c.own_off & it.own) | (c.rem_off & ~it.own);
  asm volatile("" : "+s"(it.own), "+s"(it.off));
}
DI void item_b(const Ctx& c, unsigned j, Item& it) {
  const bool in = j < c.n;
  it.need = in ? c.rem_need & ~it.own : 0u;
  it.off = in ? it.off : 0u;
  asm volatile("" : "+s"(it.need), "+s"(it.off));
}
DI void item_c(Ctx& c, const Item& it) {
  c.own_off += it.own & (14u * BLK);
  c.rem_off += ~it.own & (2u * BLK);
  asm volatile("" : "+s"(c.own_off), "+s"(c.rem_off));
}
DI void item_d(Ctx& c, const Item& it) { c.rem_need += ~it.own & 2u; asm volatile("" : "+s"(c.rem_need)); }
DI Item next_item(Ctx& c, unsigned j) { Item it; item_a(c, j, it); item_b(c, j, it); item_c(c, it); item_d(c, it); return it; }
struct Src { unsigned long long base; unsigned size, voff, step, so; };
DI void src_a(const Ctx& c, const Item& it, Src& s) {
  s.base = it.own ? c.baseX : c.baseA;
  s.size = it.own ? c.sizeX : c.sizeA;
  s.step = it.own ? 64u : 1024u;
  asm volatile("" : "+s"(s.base), "+s"(s.size), "+s"(s.step));
}
DI void src_b(const Ctx& c, const Item& it, Src& s) {
  asm volatile("v_bfi_b32 %0, %1, %2, %3" : "=v"(s.voff) : "s"(it.own), "v"(c.voffX), "v"(c.voffA));
  s.so = it.off;
}
DI __amdgpu_buffer_rsrc_t rsrc_of(const Src& s, bool on) {
  return __builtin_amdgcn_make_buffer_rsrc((void*)s.base, 0, on ? s.size : 0u, 0x00020000);
}
DI v4i ldA(__amdgpu_buffer_rsrc_t rs, Src& s) {
  const v4i r = __builtin_bit_cast(v4i, __builtin_amdgcn_raw_buffer_load_b128(rs, s.voff, s.so, A_AUX));
  s.so += s.step;
  asm volatile("" : "+s"(s.so));
  return r;
}
DI v8i cat(v4i lo, v4i hi) { return __builtin_shufflevector(lo, hi, 0, 1, 2, 3, 4, 5, 6, 7); }
DI void ld_burst(v8i (&A)[8], Src& s) {
  const __amdgpu_buffer_rsrc_t rs = rsrc_of(s, true);
#pragma unroll
  for (int k = 0; k < 6; ++k) { const v4i lo = ldA(rs, s); SB(); const v4i hi = ldA(rs, s); SB(); A[k] = cat(lo, hi); }
}
DI v8i tr_frag(LAS unsigned char* a) {
  v2i r0 = __builtin_amdgcn_ds_read_tr8_b64_v2i32((LAS v2i*)(a));
  v2i r1 = __builtin_amdgcn_ds_read_tr8_b64_v2i32((LAS v2i*)(a + 8 * 128));
  v2i r2 = __builtin_amdgcn_ds_read_tr8_b64_v2i32((LAS v2i*)(a + 64 * 128));
  v2i r3 = __builtin_amdgcn_ds_read_tr8_b64_v2i32((LAS v2i*)(a + 72 * 128));
  return (v8i){r0.x, r0.y, r1.x, r1.y, r2.x, r2.y, r3.x, r3.y};
}
DI void opaque(v8i& v) { asm volatile("" : "=v"(v)); }
DI void mfma_a0(f32x4& d, const v8i& b, const v8i& a) {
  asm volatile("v_mfma_f32_16x16x128_f8f6f4 %0, %1, %2, 0" : "=&v"(d) : "a"(b), "v"(a));
}
DI void mfma_a(f32x4& d, const v8i& b, const v8i& a) {
  asm volatile("v_mfma_f32_16x16x128_f8f6f4 %0, %1, %2, %0" : "+v"(d) : "a"(b), "v"(a));
}
DI void mfma_v(f32x4& d, const v8i& b, const v8i& a) {
  asm volatile("v_mfma_f32_16x16x128_f8f6f4 %0, %1, %2, %0" : "+v"(d) : "v"(b), "v"(a));
}
DI void add4(f32x4& a, const f32x4& b) {
#pragma unroll
  for (int e = 0; e < 4; ++e) { float x = a[e]; asm volatile("v_add_f32_e32 %0, %0, %1" : "+v"(x) : "v"(b[e])); a[e] = x; }
}
DI void mul4(f32x4& a, float s) {
#pragma unroll
  for (int e = 0; e < 4; ++e) { float x = a[e]; asm volatile("v_mul_f32_e32 %0, %1, %0" : "+v"(x) : "s"(s)); a[e] = x; }
}
DI void red_write(const Ctx& c, int cb, const f32x4& v) { *(LAS f32x4*)(size_t)(((cb & 1) ? c.wr1 : c.wr0) + (cb >> 1) * 128) = v; }
DI v4i lds_half(const Ctx& c, int l, int h) { return *(LAS v4i*)(size_t)(c.voffA + l * 2048 + h * 1024); }
DI f32x4 red_read(const Ctx& c, int h, int w) { return *(LAS f32x4*)(size_t)(c.rd + h * 256 + w * 8192); }
DI void lds_barrier() {
  asm volatile("s_waitcnt lgkmcnt(0)" ::: "memory");
  __builtin_amdgcn_s_barrier();
  asm volatile("" ::: "memory");
}
DI void out_store(const Ctx& c, unsigned off, int h, const f32x4& r) {
  __builtin_amdgcn_raw_buffer_store_b128(__builtin_bit_cast(u32x4, r), c.rsO, c.voffO, off + h * 256, OUT_AUX);
}
DI void poison(v8i (&Bv)[16]) {
#pragma unroll
  for (int cb = 0; cb < 8; ++cb) { Bv[cb][0] = 0x7f7f7f7f; asm volatile("s_nop 1" : "+v"(Bv[cb])); }
}

__global__ __launch_bounds__(256, 1) void gemm_ag_kernel(Params p) {
  __shared__ __attribute__((aligned(16))) unsigned char smem[LDS_BYTES];
  const int lane = threadIdx.x & 63;
  const int wid = __builtin_amdgcn_readfirstlane(threadIdx.x >> 6);
  const int bid = blockIdx.x;
  const unsigned rank = (unsigned)p.rank;
  if (rank >= (unsigned)NR) return;
  unsigned char* myws = p.myws;
  TS(0);
  unsigned* ctr = (unsigned*)(myws + CTR_OFF) + bid * 256 + threadIdx.x;

  if (bid < NSEND) {
    const unsigned tick = __hip_atomic_fetch_add(ctr, 1u, __ATOMIC_RELAXED, __HIP_MEMORY_SCOPE_AGENT);
    const unsigned d = (rank + (unsigned)(bid & 7)) & 7u;
    const int slice = (bid >> 3) * 4 + wid;
    unsigned char* dws = p.ws[d];
    unsigned char* dst0 = dws + DATA_OFF + (size_t)rank * SHARD + slice * 4096;
    unsigned char* flag = dws + FLAG_OFF + (rank * 16 + slice) * 128;
    if ((bid & 7) != 0) send_stream<SEND_L>(p.x + slice * 256, dst0, flag, tick, lane);
    TS(1);
    TS_OUT();
    return;
  }

  const int xcd = bid & 7, jx = (bid - NSEND) >> 3;
  constexpr int XG = 16 / XCOLS;
  const int tt = (xcd / XG) * 28 + jx;
  const int cg = (xcd % XG) * XCOLS + tt % XCOLS, g = tt / XCOLS;
  LAS unsigned char* lds = (LAS unsigned char*)smem;
  const int q = wid;
  const int g4 = lane >> 4, i16 = lane & 15;
  v8i Ba[32], Bv[16];
  unsigned tick;
  const unsigned trbase = (unsigned)(size_t)lds + q * STAGE_PER_WAVE + (16 * g4 + (i16 >> 1)) * 128 + (i16 & 1) * 8 +
                          ((((i16 >> 2) & 3) | ((g4 & 1) << 2)) * 16);
  {
    const unsigned char* wb0 = p.w + (size_t)q * 1024 * NGLOB + rank * NLOC + cg * 128;
    const unsigned sw0 = (unsigned)((lane & 7) ^ ((lane >> 4) & 3));
    const unsigned voff0 = (lane >> 3) * NGLOB + sw0 * 16, voff1 = (lane >> 3) * NGLOB + (sw0 ^ 4u) * 16;
    const unsigned stage = (unsigned)(size_t)lds + q * STAGE_PER_WAVE;
    auto dma = [&](int kb, int slot) {
      const unsigned char* wb = wb0 + (size_t)kb * 128 * NGLOB;
      unsigned dstl = stage + slot * 16384;
#pragma unroll
      for (int i = 0; i < 16; ++i) {
        asm volatile("" : "+s"(wb), "+s"(dstl));
        glds16((i & 2) ? voff1 : voff0, wb, dstl);
        wb += 8 * NGLOB; dstl += 1024;
      }
    };
    auto rd = [&](auto KB, auto SL) {
      constexpr int kb = KB.value, slot = SL.value;
      v8i o[8];
#pragma unroll
      for (int c4 = 0; c4 < 8; c4 += 4) {
        v8i t[4], o0, o1, o2, o3;
#pragma unroll
        for (int u = 0; u < 4; ++u) t[u] = tr_frag((LAS unsigned char*)(size_t)((trbase ^ ((c4 + u) * 16)) + slot * 16384));
        if constexpr (kb < 4)
          asm volatile("" : "=a"(o0), "=a"(o1), "=a"(o2), "=a"(o3) : "0"(t[0]), "1"(t[1]), "2"(t[2]), "3"(t[3]) : "memory");
        else
          asm volatile("" : "=v"(o0), "=v"(o1), "=v"(o2), "=v"(o3) : "0"(t[0]), "1"(t[1]), "2"(t[2]), "3"(t[3]) : "memory");
        o[c4] = o0; o[c4 + 1] = o1; o[c4 + 2] = o2; o[c4 + 3] = o3;
      }
#pragma unroll
      for (int cb = 0; cb < 8; ++cb) {
        if constexpr (kb < 4) Ba[kb * 8 + cb] = o[cb];
        else if constexpr (kb == 5 || kb == 6) Bv[(kb - 5) * 8 + cb] = o[cb];
        else {
          LAS v4i* d = (LAS v4i*)(size_t)(stage + lane * 16 + slot * 16384 + cb * 2048);
          d[0] = __builtin_shufflevector(o[cb], o[cb], 0, 1, 2, 3);
          d[64] = __builtin_shufflevector(o[cb], o[cb], 4, 5, 6, 7);
        }
      }
    };
    dma(0, 0); dma(1, 1);
    constexpr int LPR = XCOLS * 8, RPI = 64 / LPR, NPF = (3072 / RPI + 111) / 112;
    {
      const unsigned pvoff = (lane / LPR) * NGLOB + (lane % LPR) * 16;
      const unsigned char* pb = p.w + rank * NLOC + (xcd % XG) * (XCOLS * 128);
#pragma unroll
      for (int i = 0; i < NPF; ++i) {
        int u = (i * 112 + jx * 4 + q) * RPI; u = u < 3072 ? u : 3072 - RPI;
        asm volatile("" : "+s"(u));
        const int blk = u / 512, rr = u % 512;
        const int row = (rr >> 7) * 1024 + (2 + blk) * 128 + (rr & 127);
        glds16(pvoff, pb + (size_t)row * NGLOB, (unsigned)(size_t)lds + RED_OFF + q * 8192 + (i & 7) * 1024);
      }
    }
    waitv<16 + NPF>(); rd(IC<0>{}, IC<0>{}); dma(2, 0);
    waitv<16 + NPF>(); rd(IC<1>{}, IC<1>{}); dma(3, 1);
    waitv<16>(); rd(IC<2>{}, IC<0>{}); dma(5, 0);
    waitv<16>(); rd(IC<3>{}, IC<1>{}); dma(6, 1);
    waitv<16>(); rd(IC<5>{}, IC<0>{}); dma(4, 0);
    waitv<16>(); rd(IC<6>{}, IC<1>{}); dma(7, 1);
    tick = __hip_atomic_fetch_add(ctr, 1u, __ATOMIC_RELAXED, __HIP_MEMORY_SCOPE_AGENT);
    waitv<17>(); rd(IC<4>{}, IC<0>{});
    waitv<1>(); rd(IC<7>{}, IC<1>{});
  }

  Ctx c;
  c.call = __builtin_amdgcn_readfirstlane(tick) + 1u;
  c.dead = 0u;
  const unsigned src = (rank + 1u + (unsigned)(g >> 1)) & 7u;
  c.ownmask = (g < 4) ? (unsigned)OWNMASK : (unsigned)OWNMASK2;
  c.n = (g < 4) ? 19u : 18u;
  c.rem_off = src * SHARD + (unsigned)(g & 1) * BLK; c.rem_need = (c.call - 1u) * NBLK + (unsigned)(g & 1) + 1u;
  c.own_off = rank * SHARD + (unsigned)g * BLK;
  c.tmo = (GAS unsigned*)(myws + TMO_OFF);
  c.rsF = __builtin_amdgcn_make_buffer_rsrc(myws + FLAG_OFF + src * 2048 + q * 512, 0, 512, 0x00020000);
  c.voffA = (unsigned)(size_t)lds + q * STAGE_PER_WAVE + lane * 16;
  {
    const unsigned extra = (unsigned)(size_t)lds + q * (STAGE_PER_WAVE - 16384);
    c.baseA = (unsigned long long)(myws + DATA_OFF + (size_t)(c.call & 1u) * SLOT_BYTES - extra);
    c.sizeA = (unsigned)SLOT_BYTES + extra;
  }
  c.voffX = i16 * KDIM + g4 * 16;
  c.baseX = (unsigned long long)(p.x - (size_t)rank * SHARD + q * 1024); c.sizeX = (rank + 1u) * SHARD - q * 1024;
  c.voffF = (lane & 3) * 128;
  c.rsO = __builtin_amdgcn_make_buffer_rsrc(p.out, 0, OUT_BYTES, 0x00020000);
  c.voffO = (4 * q + g4) * (NLOC * 4) + cg * 512 + i16 * 16;
  c.wr0 = (unsigned)(size_t)lds + RED_OFF + q * 8192 + i16 * 512 + ((g4 ^ (i16 & 7)) * 16);
  c.wr1 = c.wr0 ^ 64u;
  c.rd = (unsigned)(size_t)lds + RED_OFF + (4 * q + g4) * 512 + ((i16 ^ ((4 * q + g4) & 7)) * 16);
  asm volatile("" : "+v"(c.wr1), "+v"(c.voffA), "+v"(c.voffX), "+v"(c.voffF));
  c.scale = __builtin_bit_cast(float, __builtin_amdgcn_readfirstlane(__builtin_bit_cast(unsigned, p.sx[0] * p.sw[0])));
  TS(1);

  v8i A[8];
  f32x4 acc[8];
#pragma unroll
  for (int cb = 0; cb < 8; ++cb) { acc[cb] = (f32x4){0.f, 0.f, 0.f, 0.f}; asm volatile("" : "+v"(acc[cb])); }
  Item cu = next_item(c, 0), nx = next_item(c, 1);
  if (spin(c, cu)) { c.dead = 1u; poison(Bv); }
  asm volatile("" ::: "memory");
  unsigned pollv = poll_flags(c);
  Src sr;
  src_a(c, cu, sr); src_b(c, cu, sr);
  ld_burst(A, sr);
  unsigned outp = OUT_BYTES;
  TS(2);

  for (unsigned i = 0; i < c.n; ++i) {
    f32x4 s0, s1, s2, s3, t0, t1, t2, t3;
    v8i St[2];
    v4i lo, slo;
    unsigned fl[4];
    bool ready;
    Item nn;
    __amdgpu_buffer_rsrc_t rs = rsrc_of(sr, true);
    SB();
    static_for<0, 64>([&](auto T_) {
      constexpr int t = T_.value, kb = SCH.kb[t], cb = SCH.cb[t];
      if constexpr (kb == 0) mfma_a0(acc[cb], Ba[cb], A[0]);
      else if constexpr (kb < 4) mfma_a(acc[cb], Ba[kb * 8 + cb], A[kb]);
      else if constexpr (kb == 5 || kb == 6) mfma_v(acc[cb], Bv[(kb - 5) * 8 + cb], A[kb]);
      else mfma_v(acc[cb], St[SCH.li[t] & 1], A[kb]);
      SB();
      static_for<0, 8>([&](auto C_) { if constexpr (SCH.wr[C_.value] % 64 == t) red_write(c, C_.value, acc[C_.value]); });
      static_for<0, 16>([&](auto P_) { constexpr int pc = P_.value;
        if constexpr (t == LDGAP[pc]) { if constexpr (pc & 1) A[pc / 2] = cat(lo, ldA(rs, sr)); else lo = ldA(rs, sr); } });
      if constexpr (t == 1) lanes4(pollv, fl);
      if constexpr (t == 3) ready = smin4(fl) >= nx.need;
      if constexpr (t == 7) src_a(c, nx, sr);
      if constexpr (t == 8) { src_b(c, nx, sr); rs = rsrc_of(sr, ready); }
      if constexpr (t == 28) item_a(c, i + 2, nn);
      if constexpr (t == 30) item_b(c, i + 2, nn);
      if constexpr (t == 31) item_c(c, nn);
      if constexpr (t == 33) item_d(c, nn);
      if constexpr (t == 35) pollv = poll_flags(c);
      if constexpr (t == 5) { lds_barrier(); s0 = red_read(c, 0, 0); }
      if constexpr (t == 6) { s1 = red_read(c, 0, 1); s2 = red_read(c, 0, 2); }
      if constexpr (t == 7) s3 = red_read(c, 0, 3);
      if constexpr (t == 9) add4(s0, s1);
      if constexpr (t == 10) t0 = red_read(c, 1, 0);
      if constexpr (t == 11) add4(s2, s3);
      if constexpr (t == 12) t1 = red_read(c, 1, 1);
      if constexpr (t == 13) add4(s0, s2);
      if constexpr (t == 14) t2 = red_read(c, 1, 2);
      if constexpr (t == 15) mul4(s0, c.scale);
      if constexpr (t == 16) { out_store(c, outp, 0, s0); SB(); t3 = red_read(c, 1, 3); }
      if constexpr (t == 17) add4(t0, t1);
      if constexpr (t == 19) add4(t0, t2);
      if constexpr (t == 21) add4(t0, t3);
      if constexpr (t == 22) mul4(t0, c.scale);
      if constexpr (t == 23) out_store(c, outp, 1, t0);
      if constexpr (t == 40) { asm volatile("" ::: "memory"); __builtin_amdgcn_s_barrier(); asm volatile("" ::: "memory"); }
      static_for<0, 16>([&](auto L_) { constexpr int l = L_.value;
        constexpr int at = l == 0 ? 22 : l == 1 ? 24 : LSLOT[l < 2 ? 0 : l - 2];
        if constexpr (t == at) slo = lds_half(c, l, 0);
        if constexpr (t == at + 1) St[l & 1] = cat(slo, lds_half(c, l, 1));
      });
      SB();
    });
    if (!ready) {
      if (!c.dead && spin(c, nx)) { c.dead = 1u; poison(Bv); }
      asm volatile("" ::: "memory");
      sr.so = nx.off;
      ld_burst(A, sr);
    }
    outp = 2u * cu.off;
    cu = nx; nx = nn;
    TS(3 + i);
    SB();
  }
  asm volatile("s_nop 11" : "+v"(acc[6]), "+v"(acc[7]));
  red_write(c, 6, acc[6]); red_write(c, 7, acc[7]);
  lds_barrier();
#pragma unroll
  for (int h = 0; h < 2; ++h) {
    f32x4 s0 = red_read(c, h, 0), s1 = red_read(c, h, 1), s2 = red_read(c, h, 2), s3 = red_read(c, h, 3);
    if (h == 0) { add4(s0, s1); add4(s2, s3); add4(s0, s2); } else { add4(s0, s1); add4(s0, s2); add4(s0, s3); }
    mul4(s0, c.scale);
    out_store(c, outp, h, s0);
  }
  TS(40);
  TS_OUT();
}

}

extern "C" void kernel_launch(int rank, int nranks, ncclComm_t comm,
                              hipStream_t stream,
                              void* const* d_in, const int* in_sizes, int n_in,
                              void* d_out, int out_size,
                              const int* in_dtypes, int out_dtype,
                              void* const* d_out_all,
                              void* const* d_ws_all, size_t ws_bytes) {
  (void)comm; (void)in_dtypes; (void)out_dtype; (void)d_out_all;
  if (nranks != NR || (unsigned)rank >= (unsigned)NR || n_in < 4 || ws_bytes < WS_NEED ||
      in_sizes[0] != (int)SHARD || in_sizes[1] != KDIM * NGLOB || out_size != NR * MLOC * NLOC) {
    fprintf(stderr, "kernel_launch: unexpected problem shape\n");
    return;
  }
  Params p;
  p.x = (const unsigned char*)d_in[0];
  p.w = (const unsigned char*)d_in[1];
  p.sx = (const float*)d_in[2];
  p.sw = (const float*)d_in[3];
  p.out = (float*)d_out;
  for (int r = 0; r < NR; ++r) p.ws[r] = (unsigned char*)d_ws_all[r];
  p.myws = p.ws[rank];
  p.rank = rank;
  p.pad = 0;
  hipLaunchKernelGGL(gemm_ag_kernel, dim3(NWG), dim3(256), 0, stream, p);
  HIP_CHECK(hipGetLastError());
}
```

```cpp
#include <hip/hip_runtime.h>
#include <rccl/rccl.h>
#include <cstdio>

#define HIP_CHECK(expr) do { hipError_t _e = (expr); if (_e != hipSuccess) { \
    fprintf(stderr, "HIP error %s at %s:%d\n", hipGetErrorString(_e), __FILE__, __LINE__); \
    return; } } while (0)

#define SEND_W 3
#define OWNPOS 0x248

namespace {

typedef unsigned u32x4 __attribute__((ext_vector_type(4)));
#define LAS __attribute__((address_space(3)))
#define DI __device__ __forceinline__

constexpr int NR = 8;
constexpr int KDIM = 4096, NGLOB = 16384, NLOC = 2048, MLOC = 512;
constexpr unsigned SHARD = MLOC * KDIM;
constexpr unsigned BLK = 16 * KDIM;
constexpr int NBLK = MLOC / 16;
constexpr int NWG = 256, NHELP = 32, PER_XCD = 28, XCOLS = 4;
constexpr unsigned OUT_BYTES = NR * MLOC * NLOC * 4;
constexpr unsigned FU = 64, FB = 16 * FU, FLAG_SLOT = NR * NBLK * FB;
constexpr size_t FLAG_OFF = 0;
constexpr size_t TMO_OFF = 1 << 20;
constexpr size_t TICKET_OFF = TMO_OFF + 4096;
constexpr size_t CTR_OFF = 512 << 10;
constexpr size_t DATA_OFF = 2 << 20;
constexpr size_t SLOT_BYTES = (size_t)NR * SHARD;
constexpr size_t WS_NEED = DATA_OFF + 2 * SLOT_BYTES;
constexpr unsigned long long DEADLINE_TICKS = 50000000ull;
constexpr int STAGE_PER_WAVE = 32768;
constexpr int RED_OFF = 4 * STAGE_PER_WAVE;
constexpr int LDS_BYTES = RED_OFF + 4 * 8192;

struct Params {
  const unsigned char* x;
  const unsigned char* w;
  const float* sx;
  const float* sw;
  float* out;
  unsigned char* myws;
  int rank;
  int pad;
  unsigned char* peer[NR];
};

template <int V> struct IC { static constexpr int value = V; };
template <int I, int N, class F> DI void static_for(F&& f) {
  if constexpr (I < N) { f(IC<I>{}); static_for<I + 1, N>(f); }
}
template <int N> DI void waitv() { asm volatile("s_waitcnt vmcnt(%0)" :: "i"(N) : "memory"); }
DI void pin(u32x4& v) { asm volatile("" : "+v"(v)); }
DI void ld16(u32x4& v, unsigned voff, const void* sbase) {
  asm volatile("global_load_dwordx4 %0, %1, %2" : "=v"(v) : "v"(voff), "s"(sbase) : "memory");
}
DI void st16_sys(unsigned voff, void* sbase, u32x4 v) {
  asm volatile("global_store_dwordx4 %0, %1, %2 sc0 sc1\n\ts_nop 1" :: "v"(voff), "v"(v), "s"(sbase) : "memory");
}
DI void st_flag(unsigned voff, void* sbase, unsigned v) {
  unsigned long long keep;
  asm volatile("s_mov_b64 %0, exec\n\ts_mov_b64 exec, 0xffff\n\tglobal_store_dword %1, %2, %3 sc0 sc1\n\ts_mov_b64 exec, %0"
               : "=&s"(keep) : "v"(voff), "v"(v), "s"(sbase) : "memory");
}

DI void add_one_lane(unsigned long long& old, unsigned long long* w, unsigned long long v) {
  unsigned long long keep;
  unsigned zero = 0;
  asm volatile("s_mov_b64 %1, exec\n\ts_mov_b64 exec, 1\n\tglobal_atomic_add_x2 %0, %2, %3, %4 sc0\n\ts_mov_b64 exec, %1\n\ts_nop 0"
               : "=&v"(old), "=&s"(keep) : "v"(zero), "v"(v), "s"(w) : "memory");
}
DI unsigned long long first_lane(unsigned long long v) {
  const unsigned hi = __builtin_amdgcn_readfirstlane((unsigned)(v >> 32)), lo = __builtin_amdgcn_readfirstlane((unsigned)v);
  return (unsigned long long)hi << 32 | lo;
}

constexpr int TICK = 1;
constexpr int SW = SEND_W, NST = 4 * NBLK, LAH = 8;
static_assert(SW >= 1 && SW < LAH, "load k is older than store k-7");
constexpr bool flagged(int b) { return b >= 28 || b % 2 == 1; }
struct SendPlan {
  int n[NST + SW + 1];
  constexpr SendPlan() : n() {
    int posL[NST] = {}, posS[NST] = {}, t = 0;
    for (int k = 0; k < LAH; ++k) { posL[k] = t++; if (k == TICK - 1) ++t; }
    for (int k = 0; k < NST + SW; ++k) {
      int need = -1;
      if (k < NST) need = posL[k];
      if (k >= SW && posS[k - SW] > need) need = posS[k - SW];
      n[k] = t - 1 - need;
      if (k >= SW && (k - SW) % 4 == 3 && flagged((k - SW) / 4)) ++t;
      if (k < NST) { posS[k] = t++; if (k + LAH < NST) posL[k + LAH] = t++; }
    }
  }
};
constexpr SendPlan SP;
constexpr bool send_periodic() { for (int k = 16; k < NST - 16; ++k) if (SP.n[k] != SP.n[16 + k % 8]) return false; return true; }
static_assert(send_periodic(), "steady state");

DI unsigned long long send_stream(const unsigned char* x, unsigned char* dws, unsigned rank, unsigned slice,
                                  unsigned long long* tw, unsigned long long add, bool again, int lane) {
  const unsigned char* src = x + slice * 256u;
  asm volatile("" : "+s"(src));
  const unsigned svoff = (lane & 15) * KDIM + (lane >> 4) * 16, dvoff = lane * 16, fvo = lane * 4;
  u32x4 r[LAH];
  unsigned long long old;
  static_for<0, LAH>([&](auto K_) { constexpr int k = K_.value;
    ld16(r[k], svoff, src + (k / 4) * BLK + (k % 4) * 64);
    if constexpr (k == TICK - 1) add_one_lane(old, tw, add); });
  asm volatile("" : "+s"(dws), "+s"(rank));
  unsigned char* dst0 = dws + DATA_OFF + (rank * SHARD + slice * 4096u);
  unsigned char* flag0 = dws + FLAG_OFF + (rank * (NBLK * FB) + slice * FU);
  asm volatile("s_waitcnt vmcnt(%3)" : "+v"(old), "+s"(dst0), "+s"(flag0) : "i"(LAH - TICK) : "memory");
  old = first_lane(old);
  const unsigned call = (unsigned)(old >> 35) + (again ? 0u : 1u);
  unsigned char* dst = dst0 + (size_t)(call & 1u) * SLOT_BYTES;
  unsigned char* flag = flag0 + (call & 1u) * FLAG_SLOT;
  auto one = [&](auto K_, auto N_, auto LD_, auto HEAD_, auto B0_, int b0) { constexpr int K = K_.value;
    waitv<N_.value>();
    pin(r[K % LAH]);
    if constexpr ((K + 64 - SW) % 4 == 3 && (!HEAD_.value || K >= SW) && flagged(B0_.value + (K - SW - 3) / 4)) st_flag(fvo, flag + (b0 + (K - SW - 3) / 4) * FB, call);
    st16_sys(dvoff, dst + (size_t)(b0 + K / 4) * BLK + (K % 4) * 1024, r[K % LAH]);
    if constexpr (LD_.value) ld16(r[K % LAH], svoff, src + (size_t)(b0 + K / 4 + 2) * BLK + (K % 4) * 64); };
  static_for<0, 16>([&](auto K_) { one(K_, IC<SP.n[K_.value]>{}, IC<1>{}, IC<1>{}, IC<0>{}, 0); });
  for (int b0 = 4; b0 < NBLK - 4; b0 += 2)
    static_for<0, 8>([&](auto K_) { one(K_, IC<SP.n[16 + K_.value]>{}, IC<1>{}, IC<0>{}, IC<4>{}, b0); });
  static_for<0, 16>([&](auto K_) { one(K_, IC<SP.n[NST - 16 + K_.value]>{}, IC<(K_.value < 8)>{}, IC<0>{}, IC<NBLK - 4>{}, NBLK - 4); });
  static_for<NST, NST + SW>([&](auto K_) { constexpr int k = K_.value;
    if constexpr ((k - SW) % 4 == 3) { waitv<SP.n[k]>(); st_flag(fvo, flag + ((k - SW) / 4) * FB, call); } });
  waitv<0>();
  return old;
}

static_assert(OUT_BYTES == 0x2000000 && TMO_OFF == 0x100000 && SLOT_BYTES == 0x1000000 && FLAG_SLOT == 0x40000 && FLAG_OFF == 0 &&
              DEADLINE_TICKS == 50000000 && DATA_OFF == 0x200000 && RED_OFF == 0x20000 && STAGE_PER_WAVE == 0x8000, "constants of GEMM_ASM");
#define GEMM_ASM \
  "s_mov_b32 s36, m0\n\ts_load_dword s84, %[sx], 0x0\n\ts_load_dword s85, %[sw], 0x0\n\ts_mov_b64 s[94:95], %[w]\n\ts_mov_b64 s[96:97], %[myws]\n\t" \
  "s_bfe_u32 s92, %[pk], 0x20010\n\ts_lshl_b32 s91, s92, 15\n\tv_mov_b32 v251, 1\n\tv_mov_b32 v252, 0\n\ts_mov_b64 exec, 1\n\t" \
  "global_atomic_add v250, v252, v251, %[ctr] sc0\n\ts_mov_b64 exec, -1\n\ts_lshl_b32 s78, s92, 13\n\ts_add_u32 m0, s78, 0x20000\n\ts_nop 0\n\ts_sleep 36\n\t" \
  "global_load_lds_dword %[pvoff], %[pf0]\n\tglobal_load_lds_dword %[pvoff], %[pf1]\n\tglobal_load_lds_dword %[pvoff], %[pf2]\n\ts_add_u32 s56, s94, 0x0\n\t" \
  "s_addc_u32 s57, s95, 0\n\ts_add_u32 m0, s91, 0x0\n\ts_nop 0\n\tglobal_load_lds_dwordx4 %[voff0], s[56:57]\n\ts_add_u32 m0, m0, 0x400\n\t" \
  "s_add_u32 s56, s56, 0x20000\n\ts_addc_u32 s57, s57, 0\n\tglobal_load_lds_dwordx4 %[voff0], s[56:57]\n\ts_add_u32 m0, m0, 0x400\n\t" \
  "s_add_u32 s56, s56, 0x20000\n\ts_addc_u32 s57, s57, 0\n\tglobal_load_lds_dwordx4 %[voff1], s[56:57]\n\ts_add_u32 m0, m0, 0x400\n\t" \
  "s_add_u32 s56, s56, 0x20000\n\ts_addc_u32 s57, s57, 0\n\tglobal_load_lds_dwordx4 %[voff1], s[56:57]\n\ts_add_u32 m0, m0, 0x400\n\t" \
  "s_add_u32 s56, s56, 0x20000\n\ts_addc_u32 s57, s57, 0\n\tglobal_load_lds_dwordx4 %[voff0], s[56:57]\n\ts_add_u32 m0, m0, 0x400\n\t" \
  "s_add_u32 s56, s56, 0x20000\n\ts_addc_u32 s57, s57, 0\n\tglobal_load_lds_dwordx4 %[voff0], s[56:57]\n\ts_add_u32 m0, m0, 0x400\n\t" \
  "s_add_u32 s56, s56, 0x20000\n\ts_addc_u32 s57, s57, 0\n\tglobal_load_lds_dwordx4 %[voff1], s[56:57]\n\ts_add_u32 m0, m0, 0x400\n\t" \
  "s_add_u32 s56, s56, 0x20000\n\ts_addc_u32 s57, s57, 0\n\tglobal_load_lds_dwordx4 %[voff1], s[56:57]\n\ts_add_u32 s56, s94, 0x100000\n\t" \
  "s_addc_u32 s57, s95, 0\n\ts_add_u32 m0, s91, 0x2000\n\ts_nop 0\n\tglobal_load_lds_dwordx4 %[voff0], s[56:57]\n\ts_add_u32 m0, m0, 0x400\n\t" \
  "s_add_u32 s56, s56, 0x20000\n\ts_addc_u32 s57, s57, 0\n\tglobal_load_lds_dwordx4 %[voff0], s[56:57]\n\ts_add_u32 m0, m0, 0x400\n\t" \
  "s_add_u32 s56, s56, 0x20000\n\ts_addc_u32 s57, s57, 0\n\tglobal_load_lds_dwordx4 %[voff1], s[56:57]\n\ts_add_u32 m0, m0, 0x400\n\t" \
  "s_add_u32 s56, s56, 0x20000\n\ts_addc_u32 s57, s57, 0\n\tglobal_load_lds_dwordx4 %[voff1], s[56:57]\n\ts_add_u32 m0, m0, 0x400\n\t" \
  "s_add_u32 s56, s56, 0x20000\n\ts_addc_u32 s57, s57, 0\n\tglobal_load_lds_dwordx4 %[voff0], s[56:57]\n\ts_add_u32 m0, m0, 0x400\n\t" \
  "s_add_u32 s56, s56, 0x20000\n\ts_addc_u32 s57, s57, 0\n\tglobal_load_lds_dwordx4 %[voff0], s[56:57]\n\ts_add_u32 m0, m0, 0x400\n\t" \
  "s_add_u32 s56, s56, 0x20000\n\ts_addc_u32 s57, s57, 0\n\tglobal_load_lds_dwordx4 %[voff1], s[56:57]\n\ts_add_u32 m0, m0, 0x400\n\t" \
  "s_add_u32 s56, s56, 0x20000\n\ts_addc_u32 s57, s57, 0\n\tglobal_load_lds_dwordx4 %[voff1], s[56:57]\n\ts_add_u32 s56, s94, 0x200000\n\t" \
  "s_addc_u32 s57, s95, 0\n\ts_add_u32 m0, s91, 0x4000\n\ts_nop 0\n\tglobal_load_lds_dwordx4 %[voff0], s[56:57]\n\ts_add_u32 m0, m0, 0x400\n\t" \
  "s_add_u32 s56, s56, 0x20000\n\ts_addc_u32 s57, s57, 0\n\tglobal_load_lds_dwordx4 %[voff0], s[56:57]\n\ts_add_u32 m0, m0, 0x400\n\t" \
  "s_add_u32 s56, s56, 0x20000\n\ts_addc_u32 s57, s57, 0\n\tglobal_load_lds_dwordx4 %[voff1], s[56:57]\n\ts_add_u32 m0, m0, 0x400\n\t" \
  "s_add_u32 s56, s56, 0x20000\n\ts_addc_u32 s57, s57, 0\n\tglobal_load_lds_dwordx4 %[voff1], s[56:57]\n\ts_add_u32 m0, m0, 0x400\n\t" \
  "s_add_u32 s56, s56, 0x20000\n\ts_addc_u32 s57, s57, 0\n\tglobal_load_lds_dwordx4 %[voff0], s[56:57]\n\ts_add_u32 m0, m0, 0x400\n\t" \
  "s_add_u32 s56, s56, 0x20000\n\ts_addc_u32 s57, s57, 0\n\tglobal_load_lds_dwordx4 %[voff0], s[56:57]\n\ts_add_u32 m0, m0, 0x400\n\t" \
  "s_add_u32 s56, s56, 0x20000\n\ts_addc_u32 s57, s57, 0\n\tglobal_load_lds_dwordx4 %[voff1], s[56:57]\n\ts_add_u32 m0, m0, 0x400\n\t" \
  "s_add_u32 s56, s56, 0x20000\n\ts_addc_u32 s57, s57, 0\n\tglobal_load_lds_dwordx4 %[voff1], s[56:57]\n\ts_add_u32 s56, s94, 0x300000\n\t" \
  "s_addc_u32 s57, s95, 0\n\ts_add_u32 m0, s91, 0x6000\n\ts_nop 0\n\tglobal_load_lds_dwordx4 %[voff0], s[56:57]\n\ts_add_u32 m0, m0, 0x400\n\t" \
  "s_add_u32 s56, s56, 0x20000\n\ts_addc_u32 s57, s57, 0\n\tglobal_load_lds_dwordx4 %[voff0], s[56:57]\n\ts_add_u32 m0, m0, 0x400\n\t" \
  "s_add_u32 s56, s56, 0x20000\n\ts_addc_u32 s57, s57, 0\n\tglobal_load_lds_dwordx4 %[voff1], s[56:57]\n\ts_add_u32 m0, m0, 0x400\n\t" \
  "s_add_u32 s56, s56, 0x20000\n\ts_addc_u32 s57, s57, 0\n\tglobal_load_lds_dwordx4 %[voff1], s[56:57]\n\ts_add_u32 m0, m0, 0x400\n\t" \
  "s_add_u32 s56, s56, 0x20000\n\ts_addc_u32 s57, s57, 0\n\tglobal_load_lds_dwordx4 %[voff0], s[56:57]\n\ts_add_u32 m0, m0, 0x400\n\t" \
  "s_add_u32 s56, s56, 0x20000\n\ts_addc_u32 s57, s57, 0\n\tglobal_load_lds_dwordx4 %[voff0], s[56:57]\n\ts_add_u32 m0, m0, 0x400\n\t" \
  "s_add_u32 s56, s56, 0x20000\n\ts_addc_u32 s57, s57, 0\n\tglobal_load_lds_dwordx4 %[voff1], s[56:57]\n\ts_add_u32 m0, m0, 0x400\n\t" \
  "s_add_u32 s56, s56, 0x20000\n\ts_addc_u32 s57, s57, 0\n\tglobal_load_lds_dwordx4 %[voff1], s[56:57]\n\tv_xor_b32 v240, 0, %[trb]\n\t" \
  "v_xor_b32 v241, 16, %[trb]\n\tv_xor_b32 v242, 32, %[trb]\n\tv_xor_b32 v243, 48, %[trb]\n\tv_xor_b32 v244, 64, %[trb]\n\tv_xor_b32 v245, 80, %[trb]\n\t" \
  "v_xor_b32 v246, 96, %[trb]\n\tv_xor_b32 v247, 112, %[trb]\n\ts_mov_b32 s38, 0\n\ts_mov_b32 s76, 0\n\ts_and_b32 s80, %[pk], 0xff\n\t" \
  "s_bfe_u32 s89, %[pk], 0x80008\n\ts_or_b32 s30, s80, 31\n\ts_sub_u32 s30, s30, 3\n\ts_bitcmp1_b32 %[pk], 0\n\ts_cselect_b32 s93, 0x100, 0\n\t" \
  "s_bfe_u32 s81, %[pk], 0x80014\n\ts_mov_b32 s68, 0x2000000\n\ts_mov_b32 s77, 0\n\ts_bitcmp1_b32 %[ownmask], s77\n\ts_cselect_b32 s72, -1, 0\n\t" \
  "s_cselect_b32 s69, s81, s80\n\ts_cselect_b32 s78, 0, 2\n\ts_add_u32 s80, s80, s78\n\ts_sub_u32 s81, s81, s72\n\ts_mov_b32 s87, s72\n\t" \
  "s_sub_u32 s78, s77, s89\n\ts_add_u32 s79, s30, s78\n\ts_cmp_ge_u32 s77, s89\n\ts_cselect_b32 s69, s79, s69\n\ts_mov_b32 s77, 1\n\t" \
  "s_bitcmp1_b32 %[ownmask], s77\n\ts_cselect_b32 s72, -1, 0\n\ts_cselect_b32 s70, s81, s80\n\ts_cselect_b32 s78, 0, 2\n\ts_add_u32 s80, s80, s78\n\t" \
  "s_sub_u32 s81, s81, s72\n\ts_mov_b32 s71, s72\n\ts_sub_u32 s78, s77, s89\n\ts_add_u32 s79, s30, s78\n\ts_cmp_ge_u32 s77, s89\n\t" \
  "s_cselect_b32 s70, s79, s70\n\ts_mov_b64 s[52:53], %[out]\n\ts_and_b32 s53, s53, 0xffff\n\ts_mov_b32 s54, 0x2000000\n\ts_mov_b32 s55, 0x20000\n\t" \
  "s_add_u32 s82, s96, 0x100000\n\ts_addc_u32 s83, s97, 0\n\ts_mov_b64 s[28:29], %[xb]\n\ts_and_b32 s29, s29, 0xffff\n\ts_waitcnt vmcnt(24)\n\t" \
  "v_readfirstlane_b32 s37, v250\n\ts_add_u32 s37, s37, 1\n\ts_and_b32 s78, s37, 1\n\ts_mul_i32 s79, s78, 0x1000000\n\ts_add_u32 s79, s79, 0x200000\n\t" \
  "s_lshl_b32 s77, s92, 14\n\ts_sub_u32 s79, s79, s77\n\ts_add_u32 s98, s96, s79\n\ts_addc_u32 s99, s97, 0\n\ts_and_b32 s99, s99, 0xffff\n\t" \
  "s_mov_b32 s43, 0x20000\n\ts_mov_b32 s47, 0x20000\n\ts_mul_i32 s79, s78, 0x40000\n\ts_lshl_b32 s77, s92, 8\n\ts_add_u32 s79, s79, s77\n\t" \
  "s_add_u32 s48, s96, s79\n\ts_addc_u32 s49, s97, 0\n\ts_and_b32 s49, s49, 0xffff\n\ts_sub_u32 s50, 0x40000, s77\n\ts_mov_b32 s51, 0x20000\n\t" \
  "s_waitcnt vmcnt(24)\n\tds_read_b64_tr_b8 a[0:1], v240 offset:0\n\tds_read_b64_tr_b8 a[2:3], v240 offset:1024\n\t" \
  "ds_read_b64_tr_b8 a[8:9], v241 offset:0\n\tds_read_b64_tr_b8 a[10:11], v241 offset:1024\n\tds_read_b64_tr_b8 a[16:17], v242 offset:0\n\t" \
  "ds_read_b64_tr_b8 a[18:19], v242 offset:1024\n\tds_read_b64_tr_b8 a[24:25], v243 offset:0\n\tds_read_b64_tr_b8 a[26:27], v243 offset:1024\n\t" \
  "ds_read_b64_tr_b8 a[32:33], v244 offset:0\n\tds_read_b64_tr_b8 a[34:35], v244 offset:1024\n\tds_read_b64_tr_b8 a[40:41], v245 offset:0\n\t" \
  "ds_read_b64_tr_b8 a[42:43], v245 offset:1024\n\tds_read_b64_tr_b8 a[48:49], v246 offset:0\n\tds_read_b64_tr_b8 a[50:51], v246 offset:1024\n\t" \
  "ds_read_b64_tr_b8 a[56:57], v247 offset:0\n\tds_read_b64_tr_b8 a[58:59], v247 offset:1024\n\ts_waitcnt lgkmcnt(0)\n\ts_add_u32 s56, s94, 0x400000\n\t" \
  "s_addc_u32 s57, s95, 0\n\ts_add_u32 m0, s91, 0x0\n\ts_nop 0\n\tglobal_load_lds_dwordx4 %[voff0], s[56:57]\n\ts_add_u32 m0, m0, 0x400\n\t" \
  "s_add_u32 s56, s56, 0x20000\n\ts_addc_u32 s57, s57, 0\n\tglobal_load_lds_dwordx4 %[voff0], s[56:57]\n\ts_add_u32 m0, m0, 0x400\n\t" \
  "s_add_u32 s56, s56, 0x20000\n\ts_addc_u32 s57, s57, 0\n\tglobal_load_lds_dwordx4 %[voff1], s[56:57]\n\ts_add_u32 m0, m0, 0x400\n\t" \
  "s_add_u32 s56, s56, 0x20000\n\ts_addc_u32 s57, s57, 0\n\tglobal_load_lds_dwordx4 %[voff1], s[56:57]\n\ts_add_u32 m0, m0, 0x400\n\t" \
  "s_add_u32 s56, s56, 0x20000\n\ts_addc_u32 s57, s57, 0\n\tglobal_load_lds_dwordx4 %[voff0], s[56:57]\n\ts_add_u32 m0, m0, 0x400\n\t" \
  "s_add_u32 s56, s56, 0x20000\n\ts_addc_u32 s57, s57, 0\n\tglobal_load_lds_dwordx4 %[voff0], s[56:57]\n\ts_add_u32 m0, m0, 0x400\n\t" \
  "s_add_u32 s56, s56, 0x20000\n\ts_addc_u32 s57, s57, 0\n\tglobal_load_lds_dwordx4 %[voff1], s[56:57]\n\ts_add_u32 m0, m0, 0x400\n\t" \
  "s_add_u32 s56, s56, 0x20000\n\ts_addc_u32 s57, s57, 0\n\tglobal_load_lds_dwordx4 %[voff1], s[56:57]\n\ts_waitcnt vmcnt(24)\n\t" \
  "ds_read_b64_tr_b8 a[4:5], v240 offset:8192\n\tds_read_b64_tr_b8 a[6:7], v240 offset:9216\n\tds_read_b64_tr_b8 a[12:13], v241 offset:8192\n\t" \
  "ds_read_b64_tr_b8 a[14:15], v241 offset:9216\n\tds_read_b64_tr_b8 a[20:21], v242 offset:8192\n\tds_read_b64_tr_b8 a[22:23], v242 offset:9216\n\t" \
  "ds_read_b64_tr_b8 a[28:29], v243 offset:8192\n\tds_read_b64_tr_b8 a[30:31], v243 offset:9216\n\tds_read_b64_tr_b8 a[36:37], v244 offset:8192\n\t" \
  "ds_read_b64_tr_b8 a[38:39], v244 offset:9216\n\tds_read_b64_tr_b8 a[44:45], v245 offset:8192\n\tds_read_b64_tr_b8 a[46:47], v245 offset:9216\n\t" \
  "ds_read_b64_tr_b8 a[52:53], v246 offset:8192\n\tds_read_b64_tr_b8 a[54:55], v246 offset:9216\n\tds_read_b64_tr_b8 a[60:61], v247 offset:8192\n\t" \
  "ds_read_b64_tr_b8 a[62:63], v247 offset:9216\n\ts_waitcnt lgkmcnt(0)\n\ts_add_u32 s56, s94, 0x500000\n\ts_addc_u32 s57, s95, 0\n\t" \
  "s_add_u32 m0, s91, 0x2000\n\ts_nop 0\n\tglobal_load_lds_dwordx4 %[voff0], s[56:57]\n\ts_add_u32 m0, m0, 0x400\n\ts_add_u32 s56, s56, 0x20000\n\t" \
  "s_addc_u32 s57, s57, 0\n\tglobal_load_lds_dwordx4 %[voff0], s[56:57]\n\ts_add_u32 m0, m0, 0x400\n\ts_add_u32 s56, s56, 0x20000\n\t" \
  "s_addc_u32 s57, s57, 0\n\tglobal_load_lds_dwordx4 %[voff1], s[56:57]\n\ts_add_u32 m0, m0, 0x400\n\ts_add_u32 s56, s56, 0x20000\n\t" \
  "s_addc_u32 s57, s57, 0\n\tglobal_load_lds_dwordx4 %[voff1], s[56:57]\n\ts_add_u32 m0, m0, 0x400\n\ts_add_u32 s56, s56, 0x20000\n\t" \
  "s_addc_u32 s57, s57, 0\n\tglobal_load_lds_dwordx4 %[voff0], s[56:57]\n\ts_add_u32 m0, m0, 0x400\n\ts_add_u32 s56, s56, 0x20000\n\t" \
  "s_addc_u32 s57, s57, 0\n\tglobal_load_lds_dwordx4 %[voff0], s[56:57]\n\ts_add_u32 m0, m0, 0x400\n\ts_add_u32 s56, s56, 0x20000\n\t" \
  "s_addc_u32 s57, s57, 0\n\tglobal_load_lds_dwordx4 %[voff1], s[56:57]\n\ts_add_u32 m0, m0, 0x400\n\ts_add_u32 s56, s56, 0x20000\n\t" \
  "s_addc_u32 s57, s57, 0\n\tglobal_load_lds_dwordx4 %[voff1], s[56:57]\n\ts_waitcnt vmcnt(24)\n\tds_read_b64_tr_b8 a[64:65], v240 offset:16384\n\t" \
  "ds_read_b64_tr_b8 a[66:67], v240 offset:17408\n\tds_read_b64_tr_b8 a[72:73], v241 offset:16384\n\tds_read_b64_tr_b8 a[74:75], v241 offset:17408\n\t" \
  "ds_read_b64_tr_b8 a[80:81], v242 offset:16384\n\tds_read_b64_tr_b8 a[82:83], v242 offset:17408\n\tds_read_b64_tr_b8 a[88:89], v243 offset:16384\n\t" \
  "ds_read_b64_tr_b8 a[90:91], v243 offset:17408\n\tds_read_b64_tr_b8 a[96:97], v244 offset:16384\n\tds_read_b64_tr_b8 a[98:99], v244 offset:17408\n\t" \
  "ds_read_b64_tr_b8 a[104:105], v245 offset:16384\n\tds_read_b64_tr_b8 a[106:107], v245 offset:17408\n\tds_read_b64_tr_b8 a[112:113], v246 offset:16384\n\t" \
  "ds_read_b64_tr_b8 a[114:115], v246 offset:17408\n\tds_read_b64_tr_b8 a[120:121], v247 offset:16384\n\tds_read_b64_tr_b8 a[122:123], v247 offset:17408\n\t" \
  "s_waitcnt lgkmcnt(0)\n\ts_add_u32 s56, s94, 0x600000\n\ts_addc_u32 s57, s95, 0\n\ts_add_u32 m0, s91, 0x4000\n\ts_nop 0\n\t" \
  "global_load_lds_dwordx4 %[voff0], s[56:57]\n\ts_add_u32 m0, m0, 0x400\n\ts_add_u32 s56, s56, 0x20000\n\ts_addc_u32 s57, s57, 0\n\t" \
  "global_load_lds_dwordx4 %[voff0], s[56:57]\n\ts_add_u32 m0, m0, 0x400\n\ts_add_u32 s56, s56, 0x20000\n\ts_addc_u32 s57, s57, 0\n\t" \
  "global_load_lds_dwordx4 %[voff1], s[56:57]\n\ts_add_u32 m0, m0, 0x400\n\ts_add_u32 s56, s56, 0x20000\n\ts_addc_u32 s57, s57, 0\n\t" \
  "global_load_lds_dwordx4 %[voff1], s[56:57]\n\ts_add_u32 m0, m0, 0x400\n\ts_add_u32 s56, s56, 0x20000\n\ts_addc_u32 s57, s57, 0\n\t" \
  "global_load_lds_dwordx4 %[voff0], s[56:57]\n\ts_add_u32 m0, m0, 0x400\n\ts_add_u32 s56, s56, 0x20000\n\ts_addc_u32 s57, s57, 0\n\t" \
  "global_load_lds_dwordx4 %[voff0], s[56:57]\n\ts_add_u32 m0, m0, 0x400\n\ts_add_u32 s56, s56, 0x20000\n\ts_addc_u32 s57, s57, 0\n\t" \
  "global_load_lds_dwordx4 %[voff1], s[56:57]\n\ts_add_u32 m0, m0, 0x400\n\ts_add_u32 s56, s56, 0x20000\n\ts_addc_u32 s57, s57, 0\n\t" \
  "global_load_lds_dwordx4 %[voff1], s[56:57]\n\ts_waitcnt vmcnt(24)\n\tds_read_b64_tr_b8 a[68:69], v240 offset:24576\n\t" \
  "ds_read_b64_tr_b8 a[70:71], v240 offset:25600\n\tds_read_b64_tr_b8 a[76:77], v241 offset:24576\n\tds_read_b64_tr_b8 a[78:79], v241 offset:25600\n\t" \
  "ds_read_b64_tr_b8 a[84:85], v242 offset:24576\n\tds_read_b64_tr_b8 a[86:87], v242 offset:25600\n\tds_read_b64_tr_b8 a[92:93], v243 offset:24576\n\t" \
  "ds_read_b64_tr_b8 a[94:95], v243 offset:25600\n\tds_read_b64_tr_b8 a[100:101], v244 offset:24576\n\tds_read_b64_tr_b8 a[102:103], v244 offset:25600\n\t" \
  "ds_read_b64_tr_b8 a[108:109], v245 offset:24576\n\tds_read_b64_tr_b8 a[110:111], v245 offset:25600\n\tds_read_b64_tr_b8 a[116:117], v246 offset:24576\n\t" \
  "ds_read_b64_tr_b8 a[118:119], v246 offset:25600\n\tds_read_b64_tr_b8 a[124:125], v247 offset:24576\n\tds_read_b64_tr_b8 a[126:127], v247 offset:25600\n\t" \
  "s_waitcnt lgkmcnt(0)\n\ts_add_u32 s56, s94, 0x700000\n\ts_addc_u32 s57, s95, 0\n\ts_add_u32 m0, s91, 0x6000\n\ts_nop 0\n\t" \
  "global_load_lds_dwordx4 %[voff0], s[56:57]\n\ts_add_u32 m0, m0, 0x400\n\ts_add_u32 s56, s56, 0x20000\n\ts_addc_u32 s57, s57, 0\n\t" \
  "global_load_lds_dwordx4 %[voff0], s[56:57]\n\ts_add_u32 m0, m0, 0x400\n\ts_add_u32 s56, s56, 0x20000\n\ts_addc_u32 s57, s57, 0\n\t" \
  "global_load_lds_dwordx4 %[voff1], s[56:57]\n\ts_add_u32 m0, m0, 0x400\n\ts_add_u32 s56, s56, 0x20000\n\ts_addc_u32 s57, s57, 0\n\t" \
  "global_load_lds_dwordx4 %[voff1], s[56:57]\n\ts_add_u32 m0, m0, 0x400\n\ts_add_u32 s56, s56, 0x20000\n\ts_addc_u32 s57, s57, 0\n\t" \
  "global_load_lds_dwordx4 %[voff0], s[56:57]\n\ts_add_u32 m0, m0, 0x400\n\ts_add_u32 s56, s56, 0x20000\n\ts_addc_u32 s57, s57, 0\n\t" \
  "global_load_lds_dwordx4 %[voff0], s[56:57]\n\ts_add_u32 m0, m0, 0x400\n\ts_add_u32 s56, s56, 0x20000\n\ts_addc_u32 s57, s57, 0\n\t" \
  "global_load_lds_dwordx4 %[voff1], s[56:57]\n\ts_add_u32 m0, m0, 0x400\n\ts_add_u32 s56, s56, 0x20000\n\ts_addc_u32 s57, s57, 0\n\t" \
  "global_load_lds_dwordx4 %[voff1], s[56:57]\n\ts_waitcnt vmcnt(24)\n\tds_read_b64_tr_b8 a[128:129], v240 offset:0\n\t" \
  "ds_read_b64_tr_b8 a[130:131], v240 offset:1024\n\tds_read_b64_tr_b8 a[136:137], v241 offset:0\n\tds_read_b64_tr_b8 a[138:139], v241 offset:1024\n\t" \
  "ds_read_b64_tr_b8 a[144:145], v242 offset:0\n\tds_read_b64_tr_b8 a[146:147], v242 offset:1024\n\tds_read_b64_tr_b8 a[152:153], v243 offset:0\n\t" \
  "ds_read_b64_tr_b8 a[154:155], v243 offset:1024\n\tds_read_b64_tr_b8 a[160:161], v244 offset:0\n\tds_read_b64_tr_b8 a[162:163], v244 offset:1024\n\t" \
  "ds_read_b64_tr_b8 a[168:169], v245 offset:0\n\tds_read_b64_tr_b8 a[170:171], v245 offset:1024\n\tds_read_b64_tr_b8 a[176:177], v246 offset:0\n\t" \
  "ds_read_b64_tr_b8 a[178:179], v246 offset:1024\n\tds_read_b64_tr_b8 a[184:185], v247 offset:0\n\tds_read_b64_tr_b8 a[186:187], v247 offset:1024\n\t" \
  "s_waitcnt lgkmcnt(0)\n\ts_add_u32 s56, s94, 0xa00000\n\ts_addc_u32 s57, s95, 0\n\ts_add_u32 m0, s91, 0x0\n\ts_nop 0\n\t" \
  "global_load_lds_dwordx4 %[voff0], s[56:57]\n\ts_add_u32 m0, m0, 0x400\n\ts_add_u32 s56, s56, 0x20000\n\ts_addc_u32 s57, s57, 0\n\t" \
  "global_load_lds_dwordx4 %[voff0], s[56:57]\n\ts_add_u32 m0, m0, 0x400\n\ts_add_u32 s56, s56, 0x20000\n\ts_addc_u32 s57, s57, 0\n\t" \
  "global_load_lds_dwordx4 %[voff1], s[56:57]\n\ts_add_u32 m0, m0, 0x400\n\ts_add_u32 s56, s56, 0x20000\n\ts_addc_u32 s57, s57, 0\n\t" \
  "global_load_lds_dwordx4 %[voff1], s[56:57]\n\ts_add_u32 m0, m0, 0x400\n\ts_add_u32 s56, s56, 0x20000\n\ts_addc_u32 s57, s57, 0\n\t" \
  "global_load_lds_dwordx4 %[voff0], s[56:57]\n\ts_add_u32 m0, m0, 0x400\n\ts_add_u32 s56, s56, 0x20000\n\ts_addc_u32 s57, s57, 0\n\t" \
  "global_load_lds_dwordx4 %[voff0], s[56:57]\n\ts_add_u32 m0, m0, 0x400\n\ts_add_u32 s56, s56, 0x20000\n\ts_addc_u32 s57, s57, 0\n\t" \
  "global_load_lds_dwordx4 %[voff1], s[56:57]\n\ts_add_u32 m0, m0, 0x400\n\ts_add_u32 s56, s56, 0x20000\n\ts_addc_u32 s57, s57, 0\n\t" \
  "global_load_lds_dwordx4 %[voff1], s[56:57]\n\ts_waitcnt vmcnt(24)\n\tds_read_b64_tr_b8 a[132:133], v240 offset:8192\n\t" \
  "ds_read_b64_tr_b8 a[134:135], v240 offset:9216\n\tds_read_b64_tr_b8 a[140:141], v241 offset:8192\n\tds_read_b64_tr_b8 a[142:143], v241 offset:9216\n\t" \
  "ds_read_b64_tr_b8 a[148:149], v242 offset:8192\n\tds_read_b64_tr_b8 a[150:151], v242 offset:9216\n\tds_read_b64_tr_b8 a[156:157], v243 offset:8192\n\t" \
  "ds_read_b64_tr_b8 a[158:159], v243 offset:9216\n\tds_read_b64_tr_b8 a[164:165], v244 offset:8192\n\tds_read_b64_tr_b8 a[166:167], v244 offset:9216\n\t" \
  "ds_read_b64_tr_b8 a[172:173], v245 offset:8192\n\tds_read_b64_tr_b8 a[174:175], v245 offset:9216\n\tds_read_b64_tr_b8 a[180:181], v246 offset:8192\n\t" \
  "ds_read_b64_tr_b8 a[182:183], v246 offset:9216\n\tds_read_b64_tr_b8 a[188:189], v247 offset:8192\n\tds_read_b64_tr_b8 a[190:191], v247 offset:9216\n\t" \
  "s_waitcnt lgkmcnt(0)\n\ts_add_u32 s56, s94, 0xb00000\n\ts_addc_u32 s57, s95, 0\n\ts_add_u32 m0, s91, 0x2000\n\ts_nop 0\n\t" \
  "global_load_lds_dwordx4 %[voff0], s[56:57]\n\ts_add_u32 m0, m0, 0x400\n\ts_add_u32 s56, s56, 0x20000\n\ts_addc_u32 s57, s57, 0\n\t" \
  "global_load_lds_dwordx4 %[voff0], s[56:57]\n\ts_add_u32 m0, m0, 0x400\n\ts_add_u32 s56, s56, 0x20000\n\ts_addc_u32 s57, s57, 0\n\t" \
  "global_load_lds_dwordx4 %[voff1], s[56:57]\n\ts_add_u32 m0, m0, 0x400\n\ts_add_u32 s56, s56, 0x20000\n\ts_addc_u32 s57, s57, 0\n\t" \
  "global_load_lds_dwordx4 %[voff1], s[56:57]\n\ts_add_u32 m0, m0, 0x400\n\ts_add_u32 s56, s56, 0x20000\n\ts_addc_u32 s57, s57, 0\n\t" \
  "global_load_lds_dwordx4 %[voff0], s[56:57]\n\ts_add_u32 m0, m0, 0x400\n\ts_add_u32 s56, s56, 0x20000\n\ts_addc_u32 s57, s57, 0\n\t" \
  "global_load_lds_dwordx4 %[voff0], s[56:57]\n\ts_add_u32 m0, m0, 0x400\n\ts_add_u32 s56, s56, 0x20000\n\ts_addc_u32 s57, s57, 0\n\t" \
  "global_load_lds_dwordx4 %[voff1], s[56:57]\n\ts_add_u32 m0, m0, 0x400\n\ts_add_u32 s56, s56, 0x20000\n\ts_addc_u32 s57, s57, 0\n\t" \
  "global_load_lds_dwordx4 %[voff1], s[56:57]\n\ts_waitcnt vmcnt(24)\n\tds_read_b64_tr_b8 a[192:193], v240 offset:16384\n\t" \
  "ds_read_b64_tr_b8 a[194:195], v240 offset:17408\n\tds_read_b64_tr_b8 a[200:201], v241 offset:16384\n\tds_read_b64_tr_b8 a[202:203], v241 offset:17408\n\t" \
  "ds_read_b64_tr_b8 a[208:209], v242 offset:16384\n\tds_read_b64_tr_b8 a[210:211], v242 offset:17408\n\tds_read_b64_tr_b8 a[216:217], v243 offset:16384\n\t" \
  "ds_read_b64_tr_b8 a[218:219], v243 offset:17408\n\tds_read_b64_tr_b8 a[224:225], v244 offset:16384\n\tds_read_b64_tr_b8 a[226:227], v244 offset:17408\n\t" \
  "ds_read_b64_tr_b8 a[232:233], v245 offset:16384\n\tds_read_b64_tr_b8 a[234:235], v245 offset:17408\n\tds_read_b64_tr_b8 a[240:241], v246 offset:16384\n\t" \
  "ds_read_b64_tr_b8 a[242:243], v246 offset:17408\n\tds_read_b64_tr_b8 a[248:249], v247 offset:16384\n\tds_read_b64_tr_b8 a[250:251], v247 offset:17408\n\t" \
  "s_waitcnt lgkmcnt(0)\n\ts_add_u32 s56, s94, 0xc00000\n\ts_addc_u32 s57, s95, 0\n\ts_add_u32 m0, s91, 0x4000\n\ts_nop 0\n\t" \
  "global_load_lds_dwordx4 %[voff0], s[56:57]\n\ts_add_u32 m0, m0, 0x400\n\ts_add_u32 s56, s56, 0x20000\n\ts_addc_u32 s57, s57, 0\n\t" \
  "global_load_lds_dwordx4 %[voff0], s[56:57]\n\ts_add_u32 m0, m0, 0x400\n\ts_add_u32 s56, s56, 0x20000\n\ts_addc_u32 s57, s57, 0\n\t" \
  "global_load_lds_dwordx4 %[voff1], s[56:57]\n\ts_add_u32 m0, m0, 0x400\n\ts_add_u32 s56, s56, 0x20000\n\ts_addc_u32 s57, s57, 0\n\t" \
  "global_load_lds_dwordx4 %[voff1], s[56:57]\n\ts_add_u32 m0, m0, 0x400\n\ts_add_u32 s56, s56, 0x20000\n\ts_addc_u32 s57, s57, 0\n\t" \
  "global_load_lds_dwordx4 %[voff0], s[56:57]\n\ts_add_u32 m0, m0, 0x400\n\ts_add_u32 s56, s56, 0x20000\n\ts_addc_u32 s57, s57, 0\n\t" \
  "global_load_lds_dwordx4 %[voff0], s[56:57]\n\ts_add_u32 m0, m0, 0x400\n\ts_add_u32 s56, s56, 0x20000\n\ts_addc_u32 s57, s57, 0\n\t" \
  "global_load_lds_dwordx4 %[voff1], s[56:57]\n\ts_add_u32 m0, m0, 0x400\n\ts_add_u32 s56, s56, 0x20000\n\ts_addc_u32 s57, s57, 0\n\t" \
  "global_load_lds_dwordx4 %[voff1], s[56:57]\n\ts_waitcnt vmcnt(24)\n\tds_read_b64_tr_b8 a[196:197], v240 offset:24576\n\t" \
  "ds_read_b64_tr_b8 a[198:199], v240 offset:25600\n\tds_read_b64_tr_b8 a[204:205], v241 offset:24576\n\tds_read_b64_tr_b8 a[206:207], v241 offset:25600\n\t" \
  "ds_read_b64_tr_b8 a[212:213], v242 offset:24576\n\tds_read_b64_tr_b8 a[214:215], v242 offset:25600\n\tds_read_b64_tr_b8 a[220:221], v243 offset:24576\n\t" \
  "ds_read_b64_tr_b8 a[222:223], v243 offset:25600\n\tds_read_b64_tr_b8 a[228:229], v244 offset:24576\n\tds_read_b64_tr_b8 a[230:231], v244 offset:25600\n\t" \
  "ds_read_b64_tr_b8 a[236:237], v245 offset:24576\n\tds_read_b64_tr_b8 a[238:239], v245 offset:25600\n\tds_read_b64_tr_b8 a[244:245], v246 offset:24576\n\t" \
  "ds_read_b64_tr_b8 a[246:247], v246 offset:25600\n\tds_read_b64_tr_b8 a[252:253], v247 offset:24576\n\tds_read_b64_tr_b8 a[254:255], v247 offset:25600\n\t" \
  "s_waitcnt lgkmcnt(0)\n\ts_add_u32 s56, s94, 0xd00000\n\ts_addc_u32 s57, s95, 0\n\ts_add_u32 m0, s91, 0x6000\n\ts_nop 0\n\t" \
  "global_load_lds_dwordx4 %[voff0], s[56:57]\n\ts_add_u32 m0, m0, 0x400\n\ts_add_u32 s56, s56, 0x20000\n\ts_addc_u32 s57, s57, 0\n\t" \
  "global_load_lds_dwordx4 %[voff0], s[56:57]\n\ts_add_u32 m0, m0, 0x400\n\ts_add_u32 s56, s56, 0x20000\n\ts_addc_u32 s57, s57, 0\n\t" \
  "global_load_lds_dwordx4 %[voff1], s[56:57]\n\ts_add_u32 m0, m0, 0x400\n\ts_add_u32 s56, s56, 0x20000\n\ts_addc_u32 s57, s57, 0\n\t" \
  "global_load_lds_dwordx4 %[voff1], s[56:57]\n\ts_add_u32 m0, m0, 0x400\n\ts_add_u32 s56, s56, 0x20000\n\ts_addc_u32 s57, s57, 0\n\t" \
  "global_load_lds_dwordx4 %[voff0], s[56:57]\n\ts_add_u32 m0, m0, 0x400\n\ts_add_u32 s56, s56, 0x20000\n\ts_addc_u32 s57, s57, 0\n\t" \
  "global_load_lds_dwordx4 %[voff0], s[56:57]\n\ts_add_u32 m0, m0, 0x400\n\ts_add_u32 s56, s56, 0x20000\n\ts_addc_u32 s57, s57, 0\n\t" \
  "global_load_lds_dwordx4 %[voff1], s[56:57]\n\ts_add_u32 m0, m0, 0x400\n\ts_add_u32 s56, s56, 0x20000\n\ts_addc_u32 s57, s57, 0\n\t" \
  "global_load_lds_dwordx4 %[voff1], s[56:57]\n\ts_waitcnt vmcnt(24)\n\tds_read_b64_tr_b8 v[16:17], v240 offset:0\n\t" \
  "ds_read_b64_tr_b8 v[18:19], v240 offset:1024\n\tds_read_b64_tr_b8 v[24:25], v241 offset:0\n\tds_read_b64_tr_b8 v[26:27], v241 offset:1024\n\t" \
  "ds_read_b64_tr_b8 v[32:33], v242 offset:0\n\tds_read_b64_tr_b8 v[34:35], v242 offset:1024\n\tds_read_b64_tr_b8 v[40:41], v243 offset:0\n\t" \
  "ds_read_b64_tr_b8 v[42:43], v243 offset:1024\n\tds_read_b64_tr_b8 v[48:49], v244 offset:0\n\tds_read_b64_tr_b8 v[50:51], v244 offset:1024\n\t" \
  "ds_read_b64_tr_b8 v[56:57], v245 offset:0\n\tds_read_b64_tr_b8 v[58:59], v245 offset:1024\n\tds_read_b64_tr_b8 v[64:65], v246 offset:0\n\t" \
  "ds_read_b64_tr_b8 v[66:67], v246 offset:1024\n\tds_read_b64_tr_b8 v[72:73], v247 offset:0\n\tds_read_b64_tr_b8 v[74:75], v247 offset:1024\n\t" \
  "s_waitcnt lgkmcnt(0)\n\ts_add_u32 s56, s94, 0x800000\n\ts_addc_u32 s57, s95, 0\n\ts_add_u32 m0, s91, 0x0\n\ts_nop 0\n\t" \
  "global_load_lds_dwordx4 %[voff0], s[56:57]\n\ts_add_u32 m0, m0, 0x400\n\ts_add_u32 s56, s56, 0x20000\n\ts_addc_u32 s57, s57, 0\n\t" \
  "global_load_lds_dwordx4 %[voff0], s[56:57]\n\ts_add_u32 m0, m0, 0x400\n\ts_add_u32 s56, s56, 0x20000\n\ts_addc_u32 s57, s57, 0\n\t" \
  "global_load_lds_dwordx4 %[voff1], s[56:57]\n\ts_add_u32 m0, m0, 0x400\n\ts_add_u32 s56, s56, 0x20000\n\ts_addc_u32 s57, s57, 0\n\t" \
  "global_load_lds_dwordx4 %[voff1], s[56:57]\n\ts_add_u32 m0, m0, 0x400\n\ts_add_u32 s56, s56, 0x20000\n\ts_addc_u32 s57, s57, 0\n\t" \
  "global_load_lds_dwordx4 %[voff0], s[56:57]\n\ts_add_u32 m0, m0, 0x400\n\ts_add_u32 s56, s56, 0x20000\n\ts_addc_u32 s57, s57, 0\n\t" \
  "global_load_lds_dwordx4 %[voff0], s[56:57]\n\ts_add_u32 m0, m0, 0x400\n\ts_add_u32 s56, s56, 0x20000\n\ts_addc_u32 s57, s57, 0\n\t" \
  "global_load_lds_dwordx4 %[voff1], s[56:57]\n\ts_add_u32 m0, m0, 0x400\n\ts_add_u32 s56, s56, 0x20000\n\ts_addc_u32 s57, s57, 0\n\t" \
  "global_load_lds_dwordx4 %[voff1], s[56:57]\n\ts_and_b32 s88, s69, 31\n\ts_cmp_lt_u32 s88, 28\n\ts_cselect_b32 s88, 1, 0\n\ts_or_b32 s88, s88, s69\n\t" \
  "s_lshl_b32 s88, s88, 10\n\tbuffer_load_dword v248, %[voffF], s[48:51], s88 offen sc0 sc1\n\ts_and_b32 s88, s70, 31\n\ts_cmp_lt_u32 s88, 28\n\t" \
  "s_cselect_b32 s88, 1, 0\n\ts_or_b32 s88, s88, s70\n\ts_lshl_b32 s88, s88, 10\n\tbuffer_load_dword v249, %[voffF], s[48:51], s88 offen sc0 sc1\n\t" \
  "s_waitcnt vmcnt(24)\n\tds_read_b64_tr_b8 v[20:21], v240 offset:8192\n\tds_read_b64_tr_b8 v[22:23], v240 offset:9216\n\t" \
  "ds_read_b64_tr_b8 v[28:29], v241 offset:8192\n\tds_read_b64_tr_b8 v[30:31], v241 offset:9216\n\tds_read_b64_tr_b8 v[36:37], v242 offset:8192\n\t" \
  "ds_read_b64_tr_b8 v[38:39], v242 offset:9216\n\tds_read_b64_tr_b8 v[44:45], v243 offset:8192\n\tds_read_b64_tr_b8 v[46:47], v243 offset:9216\n\t" \
  "ds_read_b64_tr_b8 v[52:53], v244 offset:8192\n\tds_read_b64_tr_b8 v[54:55], v244 offset:9216\n\tds_read_b64_tr_b8 v[60:61], v245 offset:8192\n\t" \
  "ds_read_b64_tr_b8 v[62:63], v245 offset:9216\n\tds_read_b64_tr_b8 v[68:69], v246 offset:8192\n\tds_read_b64_tr_b8 v[70:71], v246 offset:9216\n\t" \
  "ds_read_b64_tr_b8 v[76:77], v247 offset:8192\n\tds_read_b64_tr_b8 v[78:79], v247 offset:9216\n\ts_waitcnt lgkmcnt(0)\n\ts_add_u32 s56, s94, 0x900000\n\t" \
  "s_addc_u32 s57, s95, 0\n\ts_add_u32 m0, s91, 0x2000\n\ts_nop 0\n\tglobal_load_lds_dwordx4 %[voff0], s[56:57]\n\ts_add_u32 m0, m0, 0x400\n\t" \
  "s_add_u32 s56, s56, 0x20000\n\ts_addc_u32 s57, s57, 0\n\tglobal_load_lds_dwordx4 %[voff0], s[56:57]\n\ts_add_u32 m0, m0, 0x400\n\t" \
  "s_add_u32 s56, s56, 0x20000\n\ts_addc_u32 s57, s57, 0\n\tglobal_load_lds_dwordx4 %[voff1], s[56:57]\n\ts_add_u32 m0, m0, 0x400\n\t" \
  "s_add_u32 s56, s56, 0x20000\n\ts_addc_u32 s57, s57, 0\n\tglobal_load_lds_dwordx4 %[voff1], s[56:57]\n\ts_add_u32 m0, m0, 0x400\n\t" \
  "s_add_u32 s56, s56, 0x20000\n\ts_addc_u32 s57, s57, 0\n\tglobal_load_lds_dwordx4 %[voff0], s[56:57]\n\ts_add_u32 m0, m0, 0x400\n\t" \
  "s_add_u32 s56, s56, 0x20000\n\ts_addc_u32 s57, s57, 0\n\tglobal_load_lds_dwordx4 %[voff0], s[56:57]\n\ts_add_u32 m0, m0, 0x400\n\t" \
  "s_add_u32 s56, s56, 0x20000\n\ts_addc_u32 s57, s57, 0\n\tglobal_load_lds_dwordx4 %[voff1], s[56:57]\n\ts_add_u32 m0, m0, 0x400\n\t" \
  "s_add_u32 s56, s56, 0x20000\n\ts_addc_u32 s57, s57, 0\n\tglobal_load_lds_dwordx4 %[voff1], s[56:57]\n\ts_waitcnt vmcnt(24)\n\t" \
  "ds_read_b64_tr_b8 v[80:81], v240 offset:16384\n\tds_read_b64_tr_b8 v[82:83], v240 offset:17408\n\tds_read_b64_tr_b8 v[88:89], v241 offset:16384\n\t" \
  "ds_read_b64_tr_b8 v[90:91], v241 offset:17408\n\tds_read_b64_tr_b8 v[96:97], v242 offset:16384\n\tds_read_b64_tr_b8 v[98:99], v242 offset:17408\n\t" \
  "ds_read_b64_tr_b8 v[104:105], v243 offset:16384\n\tds_read_b64_tr_b8 v[106:107], v243 offset:17408\n\tds_read_b64_tr_b8 v[112:113], v244 offset:16384\n\t" \
  "ds_read_b64_tr_b8 v[114:115], v244 offset:17408\n\tds_read_b64_tr_b8 v[120:121], v245 offset:16384\n\tds_read_b64_tr_b8 v[122:123], v245 offset:17408\n\t" \
  "ds_read_b64_tr_b8 v[128:129], v246 offset:16384\n\tds_read_b64_tr_b8 v[130:131], v246 offset:17408\n\tds_read_b64_tr_b8 v[136:137], v247 offset:16384\n\t" \
  "ds_read_b64_tr_b8 v[138:139], v247 offset:17408\n\ts_waitcnt lgkmcnt(0)\n\ts_add_u32 s56, s94, 0xe00000\n\ts_addc_u32 s57, s95, 0\n\t" \
  "s_add_u32 m0, s91, 0x4000\n\ts_nop 0\n\tglobal_load_lds_dwordx4 %[voff0], s[56:57]\n\ts_add_u32 m0, m0, 0x400\n\ts_add_u32 s56, s56, 0x20000\n\t" \
  "s_addc_u32 s57, s57, 0\n\tglobal_load_lds_dwordx4 %[voff0], s[56:57]\n\ts_add_u32 m0, m0, 0x400\n\ts_add_u32 s56, s56, 0x20000\n\t" \
  "s_addc_u32 s57, s57, 0\n\tglobal_load_lds_dwordx4 %[voff1], s[56:57]\n\ts_add_u32 m0, m0, 0x400\n\ts_add_u32 s56, s56, 0x20000\n\t" \
  "s_addc_u32 s57, s57, 0\n\tglobal_load_lds_dwordx4 %[voff1], s[56:57]\n\ts_add_u32 m0, m0, 0x400\n\ts_add_u32 s56, s56, 0x20000\n\t" \
  "s_addc_u32 s57, s57, 0\n\tglobal_load_lds_dwordx4 %[voff0], s[56:57]\n\ts_add_u32 m0, m0, 0x400\n\ts_add_u32 s56, s56, 0x20000\n\t" \
  "s_addc_u32 s57, s57, 0\n\tglobal_load_lds_dwordx4 %[voff0], s[56:57]\n\ts_add_u32 m0, m0, 0x400\n\ts_add_u32 s56, s56, 0x20000\n\t" \
  "s_addc_u32 s57, s57, 0\n\tglobal_load_lds_dwordx4 %[voff1], s[56:57]\n\ts_add_u32 m0, m0, 0x400\n\ts_add_u32 s56, s56, 0x20000\n\t" \
  "s_addc_u32 s57, s57, 0\n\tglobal_load_lds_dwordx4 %[voff1], s[56:57]\n\ts_waitcnt vmcnt(24)\n\tds_read_b64_tr_b8 v[84:85], v240 offset:24576\n\t" \
  "ds_read_b64_tr_b8 v[86:87], v240 offset:25600\n\tds_read_b64_tr_b8 v[92:93], v241 offset:24576\n\tds_read_b64_tr_b8 v[94:95], v241 offset:25600\n\t" \
  "ds_read_b64_tr_b8 v[100:101], v242 offset:24576\n\tds_read_b64_tr_b8 v[102:103], v242 offset:25600\n\tds_read_b64_tr_b8 v[108:109], v243 offset:24576\n\t" \
  "ds_read_b64_tr_b8 v[110:111], v243 offset:25600\n\tds_read_b64_tr_b8 v[116:117], v244 offset:24576\n\tds_read_b64_tr_b8 v[118:119], v244 offset:25600\n\t" \
  "ds_read_b64_tr_b8 v[124:125], v245 offset:24576\n\tds_read_b64_tr_b8 v[126:127], v245 offset:25600\n\tds_read_b64_tr_b8 v[132:133], v246 offset:24576\n\t" \
  "ds_read_b64_tr_b8 v[134:135], v246 offset:25600\n\tds_read_b64_tr_b8 v[140:141], v247 offset:24576\n\tds_read_b64_tr_b8 v[142:143], v247 offset:25600\n\t" \
  "s_waitcnt lgkmcnt(0)\n\ts_add_u32 s56, s94, 0xf00000\n\ts_addc_u32 s57, s95, 0\n\ts_add_u32 m0, s91, 0x6000\n\ts_nop 0\n\t" \
  "global_load_lds_dwordx4 %[voff0], s[56:57]\n\ts_add_u32 m0, m0, 0x400\n\ts_add_u32 s56, s56, 0x20000\n\ts_addc_u32 s57, s57, 0\n\t" \
  "global_load_lds_dwordx4 %[voff0], s[56:57]\n\ts_add_u32 m0, m0, 0x400\n\ts_add_u32 s56, s56, 0x20000\n\ts_addc_u32 s57, s57, 0\n\t" \
  "global_load_lds_dwordx4 %[voff1], s[56:57]\n\ts_add_u32 m0, m0, 0x400\n\ts_add_u32 s56, s56, 0x20000\n\ts_addc_u32 s57, s57, 0\n\t" \
  "global_load_lds_dwordx4 %[voff1], s[56:57]\n\ts_add_u32 m0, m0, 0x400\n\ts_add_u32 s56, s56, 0x20000\n\ts_addc_u32 s57, s57, 0\n\t" \
  "global_load_lds_dwordx4 %[voff0], s[56:57]\n\ts_add_u32 m0, m0, 0x400\n\ts_add_u32 s56, s56, 0x20000\n\ts_addc_u32 s57, s57, 0\n\t" \
  "global_load_lds_dwordx4 %[voff0], s[56:57]\n\ts_add_u32 m0, m0, 0x400\n\ts_add_u32 s56, s56, 0x20000\n\ts_addc_u32 s57, s57, 0\n\t" \
  "global_load_lds_dwordx4 %[voff1], s[56:57]\n\ts_add_u32 m0, m0, 0x400\n\ts_add_u32 s56, s56, 0x20000\n\ts_addc_u32 s57, s57, 0\n\t" \
  "global_load_lds_dwordx4 %[voff1], s[56:57]\n\ts_waitcnt vmcnt(24)\n\tds_read_b64_tr_b8 v[208:209], v240 offset:0\n\t" \
  "ds_read_b64_tr_b8 v[210:211], v240 offset:1024\n\tds_read_b64_tr_b8 v[212:213], v241 offset:0\n\tds_read_b64_tr_b8 v[214:215], v241 offset:1024\n\t" \
  "ds_read_b64_tr_b8 v[216:217], v242 offset:0\n\tds_read_b64_tr_b8 v[218:219], v242 offset:1024\n\tds_read_b64_tr_b8 v[220:221], v243 offset:0\n\t" \
  "ds_read_b64_tr_b8 v[222:223], v243 offset:1024\n\tds_read_b64_tr_b8 v[224:225], v244 offset:0\n\tds_read_b64_tr_b8 v[226:227], v244 offset:1024\n\t" \
  "ds_read_b64_tr_b8 v[228:229], v245 offset:0\n\tds_read_b64_tr_b8 v[230:231], v245 offset:1024\n\tds_read_b64_tr_b8 v[232:233], v246 offset:0\n\t" \
  "ds_read_b64_tr_b8 v[234:235], v246 offset:1024\n\tds_read_b64_tr_b8 v[236:237], v247 offset:0\n\tds_read_b64_tr_b8 v[238:239], v247 offset:1024\n\t" \
  "s_waitcnt lgkmcnt(0)\n\tds_write_b128 %[voffA], v[208:211] offset:0\n\tds_write_b128 %[voffA], v[212:215] offset:1024\n\t" \
  "ds_write_b128 %[voffA], v[216:219] offset:2048\n\tds_write_b128 %[voffA], v[220:223] offset:3072\n\tds_write_b128 %[voffA], v[224:227] offset:4096\n\t" \
  "ds_write_b128 %[voffA], v[228:231] offset:5120\n\tds_write_b128 %[voffA], v[232:235] offset:6144\n\tds_write_b128 %[voffA], v[236:239] offset:7168\n\t" \
  "s_waitcnt vmcnt(16)\n\tv_readlane_b32 s62, v248, 0\n\tv_readlane_b32 s63, v248, 1\n\tv_readlane_b32 s64, v248, 2\n\tv_readlane_b32 s65, v248, 3\n\t" \
  "s_min_u32 s62, s62, s63\n\ts_min_u32 s64, s64, s65\n\ts_min_u32 s62, s62, s64\n\ts_or_b32 s62, s62, s87\n\ts_cmp_ge_u32 s62, s37\n\t" \
  "s_cselect_b32 s66, 1, 0\n\tv_mov_b32 %[voff1], v249\n\ts_mov_b32 s95, %[xsz]\n\ts_lshl_b32 s94, s92, 14\n\ts_add_u32 s94, s94, 0x1000000\n\t" \
  "s_cmp_lg_u32 s87, 0\n\ts_cselect_b64 s[40:41], s[28:29], s[98:99]\n\ts_cselect_b32 s42, s95, s94\n\ts_cselect_b32 s90, 64, 0x400\n\t" \
  "v_bfi_b32 %[trb], s87, %[voffX], %[voffA]\n\ts_mov_b64 s[44:45], s[40:41]\n\ts_lshl_b32 s67, s69, 16\n\ts_cmp_lg_u32 s66, 0\n\t" \
  "s_cselect_b32 s46, s42, 0\n\tbuffer_load_dwordx4 v[144:147], %[trb], s[44:47], s67 offen sc0 sc1\n\ts_add_u32 s67, s67, s90\n\t" \
  "buffer_load_dwordx4 v[148:151], %[trb], s[44:47], s67 offen sc0 sc1\n\ts_add_u32 s67, s67, s90\n\t" \
  "buffer_load_dwordx4 v[152:155], %[trb], s[44:47], s67 offen sc0 sc1\n\ts_add_u32 s67, s67, s90\n\t" \
  "buffer_load_dwordx4 v[156:159], %[trb], s[44:47], s67 offen sc0 sc1\n\ts_add_u32 s67, s67, s90\n\t" \
  "buffer_load_dwordx4 v[160:163], %[trb], s[44:47], s67 offen sc0 sc1\n\ts_add_u32 s67, s67, s90\n\t" \
  "buffer_load_dwordx4 v[164:167], %[trb], s[44:47], s67 offen sc0 sc1\n\ts_add_u32 s67, s67, s90\n\t" \
  "buffer_load_dwordx4 v[168:171], %[trb], s[44:47], s67 offen sc0 sc1\n\ts_add_u32 s67, s67, s90\n\t" \
  "buffer_load_dwordx4 v[172:175], %[trb], s[44:47], s67 offen sc0 sc1\n\ts_add_u32 s67, s67, s90\n\t" \
  "buffer_load_dwordx4 v[176:179], %[trb], s[44:47], s67 offen sc0 sc1\n\ts_add_u32 s67, s67, s90\n\t" \
  "buffer_load_dwordx4 v[180:183], %[trb], s[44:47], s67 offen sc0 sc1\n\ts_add_u32 s67, s67, s90\n\t" \
  "buffer_load_dwordx4 v[184:187], %[trb], s[44:47], s67 offen sc0 sc1\n\ts_add_u32 s67, s67, s90\n\t" \
  "buffer_load_dwordx4 v[188:191], %[trb], s[44:47], s67 offen sc0 sc1\n\ts_add_u32 s67, s67, s90\n\tds_read_b64_tr_b8 v[208:209], v240 offset:8192\n\t" \
  "ds_read_b64_tr_b8 v[210:211], v240 offset:9216\n\tds_read_b64_tr_b8 v[212:213], v241 offset:8192\n\tds_read_b64_tr_b8 v[214:215], v241 offset:9216\n\t" \
  "ds_read_b64_tr_b8 v[216:217], v242 offset:8192\n\tds_read_b64_tr_b8 v[218:219], v242 offset:9216\n\tds_read_b64_tr_b8 v[220:221], v243 offset:8192\n\t" \
  "ds_read_b64_tr_b8 v[222:223], v243 offset:9216\n\tds_read_b64_tr_b8 v[224:225], v244 offset:8192\n\tds_read_b64_tr_b8 v[226:227], v244 offset:9216\n\t" \
  "ds_read_b64_tr_b8 v[228:229], v245 offset:8192\n\tds_read_b64_tr_b8 v[230:231], v245 offset:9216\n\tds_read_b64_tr_b8 v[232:233], v246 offset:8192\n\t" \
  "ds_read_b64_tr_b8 v[234:235], v246 offset:9216\n\tds_read_b64_tr_b8 v[236:237], v247 offset:8192\n\tds_read_b64_tr_b8 v[238:239], v247 offset:9216\n\t" \
  "s_waitcnt lgkmcnt(0)\n\tds_write_b128 %[voffA], v[208:211] offset:8192\n\tds_write_b128 %[voffA], v[212:215] offset:9216\n\t" \
  "ds_write_b128 %[voffA], v[216:219] offset:10240\n\tds_write_b128 %[voffA], v[220:223] offset:11264\n\tds_write_b128 %[voffA], v[224:227] offset:12288\n\t" \
  "ds_write_b128 %[voffA], v[228:231] offset:13312\n\tds_write_b128 %[voffA], v[232:235] offset:14336\n\tds_write_b128 %[voffA], v[236:239] offset:15360\n\t" \
  "s_waitcnt vmcnt(20)\n\tds_read_b64_tr_b8 v[208:209], v240 offset:16384\n\tds_read_b64_tr_b8 v[210:211], v240 offset:17408\n\t" \
  "ds_read_b64_tr_b8 v[212:213], v241 offset:16384\n\tds_read_b64_tr_b8 v[214:215], v241 offset:17408\n\tds_read_b64_tr_b8 v[216:217], v242 offset:16384\n\t" \
  "ds_read_b64_tr_b8 v[218:219], v242 offset:17408\n\tds_read_b64_tr_b8 v[220:221], v243 offset:16384\n\tds_read_b64_tr_b8 v[222:223], v243 offset:17408\n\t" \
  "ds_read_b64_tr_b8 v[224:225], v244 offset:16384\n\tds_read_b64_tr_b8 v[226:227], v244 offset:17408\n\tds_read_b64_tr_b8 v[228:229], v245 offset:16384\n\t" \
  "ds_read_b64_tr_b8 v[230:231], v245 offset:17408\n\tds_read_b64_tr_b8 v[232:233], v246 offset:16384\n\tds_read_b64_tr_b8 v[234:235], v246 offset:17408\n\t" \
  "ds_read_b64_tr_b8 v[236:237], v247 offset:16384\n\tds_read_b64_tr_b8 v[238:239], v247 offset:17408\n\ts_waitcnt lgkmcnt(0)\n\t" \
  "ds_write_b128 %[voffA], v[208:211] offset:16384\n\tds_write_b128 %[voffA], v[212:215] offset:17408\n\tds_write_b128 %[voffA], v[216:219] offset:18432\n\t" \
  "ds_write_b128 %[voffA], v[220:223] offset:19456\n\tds_write_b128 %[voffA], v[224:227] offset:20480\n\tds_write_b128 %[voffA], v[228:231] offset:21504\n\t" \
  "ds_write_b128 %[voffA], v[232:235] offset:22528\n\tds_write_b128 %[voffA], v[236:239] offset:23552\n\ts_waitcnt vmcnt(12)\n\t" \
  "ds_read_b64_tr_b8 v[208:209], v240 offset:24576\n\tds_read_b64_tr_b8 v[210:211], v240 offset:25600\n\tds_read_b64_tr_b8 v[212:213], v241 offset:24576\n\t" \
  "ds_read_b64_tr_b8 v[214:215], v241 offset:25600\n\tds_read_b64_tr_b8 v[216:217], v242 offset:24576\n\tds_read_b64_tr_b8 v[218:219], v242 offset:25600\n\t" \
  "ds_read_b64_tr_b8 v[220:221], v243 offset:24576\n\tds_read_b64_tr_b8 v[222:223], v243 offset:25600\n\tds_read_b64_tr_b8 v[224:225], v244 offset:24576\n\t" \
  "ds_read_b64_tr_b8 v[226:227], v244 offset:25600\n\tds_read_b64_tr_b8 v[228:229], v245 offset:24576\n\tds_read_b64_tr_b8 v[230:231], v245 offset:25600\n\t" \
  "ds_read_b64_tr_b8 v[232:233], v246 offset:24576\n\tds_read_b64_tr_b8 v[234:235], v246 offset:25600\n\tds_read_b64_tr_b8 v[236:237], v247 offset:24576\n\t" \
  "ds_read_b64_tr_b8 v[238:239], v247 offset:25600\n\ts_waitcnt lgkmcnt(0)\n\tds_write_b128 %[voffA], v[208:211] offset:24576\n\t" \
  "ds_write_b128 %[voffA], v[212:215] offset:25600\n\tds_write_b128 %[voffA], v[216:219] offset:26624\n\tds_write_b128 %[voffA], v[220:223] offset:27648\n\t" \
  "ds_write_b128 %[voffA], v[224:227] offset:28672\n\tds_write_b128 %[voffA], v[228:231] offset:29696\n\tds_write_b128 %[voffA], v[232:235] offset:30720\n\t" \
  "ds_write_b128 %[voffA], v[236:239] offset:31744\n\tv_mov_b32 %[pvoff], s84\n\tv_mul_f32_e32 %[pvoff], s85, %[pvoff]\n\ts_nop 0\n\t" \
  "v_readfirstlane_b32 s39, %[pvoff]\n\ts_cmp_lg_u32 s66, 0\n\ts_cbranch_scc1 Lloop_%=\n\ts_cmp_lg_u32 s38, 0\n\ts_cbranch_scc1 LgoA_%=\n\t" \
  "s_memrealtime s[58:59]\n\ts_mov_b32 s77, 0\n\t" \
  "LspinA_%=:\ns_and_b32 s88, s69, 31\n\ts_cmp_lt_u32 s88, 28\n\ts_cselect_b32 s88, 1, 0\n\ts_or_b32 s88, s88, s69\n\ts_lshl_b32 s88, s88, 10\n\t" \
  "buffer_load_dword %[voff0], %[voffF], s[48:51], s88 offen sc0 sc1\n\ts_waitcnt vmcnt(0)\n\tv_readlane_b32 s62, %[voff0], 0\n\t" \
  "v_readlane_b32 s63, %[voff0], 1\n\tv_readlane_b32 s64, %[voff0], 2\n\tv_readlane_b32 s65, %[voff0], 3\n\ts_min_u32 s62, s62, s63\n\t" \
  "s_min_u32 s64, s64, s65\n\ts_min_u32 s62, s62, s64\n\ts_or_b32 s62, s62, s87\n\ts_cmp_ge_u32 s62, s37\n\ts_cbranch_scc1 LgoA_%=\n\ts_sleep 2\n\t" \
  "s_add_u32 s77, s77, 1\n\ts_and_b32 s78, s77, 63\n\ts_cbranch_scc1 LspinA_%=\n\tv_mov_b32 %[pvoff], 0\n\t" \
  "global_load_dword %[voff0], %[pvoff], s[82:83] sc1\n\ts_memrealtime s[60:61]\n\ts_waitcnt vmcnt(0) lgkmcnt(0)\n\tv_readfirstlane_b32 s78, %[voff0]\n\t" \
  "s_cmp_eq_u32 s78, s37\n\ts_cbranch_scc1 LtmoA_%=\n\ts_sub_u32 s60, s60, s58\n\ts_subb_u32 s61, s61, s59\n\ts_cmp_lg_u32 s61, 0\n\t" \
  "s_cbranch_scc1 LtmoA_%=\n\ts_cmp_le_u32 s60, 50000000\n\ts_cbranch_scc1 LspinA_%=\n\t" \
  "LtmoA_%=:\nv_mov_b32 %[voff0], s37\n\tglobal_store_dword %[pvoff], %[voff0], s[82:83] sc1\n\ts_mov_b32 s38, 1\n\tv_mov_b32 v16, 0x7f7f7f7f\n\t" \
  "v_mov_b32 v24, 0x7f7f7f7f\n\tv_mov_b32 v32, 0x7f7f7f7f\n\tv_mov_b32 v40, 0x7f7f7f7f\n\tv_mov_b32 v48, 0x7f7f7f7f\n\tv_mov_b32 v56, 0x7f7f7f7f\n\t" \
  "v_mov_b32 v64, 0x7f7f7f7f\n\tv_mov_b32 v72, 0x7f7f7f7f\n\ts_waitcnt vmcnt(0)\n\t" \
  "LgoA_%=:\ns_lshl_b32 s67, s69, 16\n\tbuffer_load_dwordx4 v[144:147], %[trb], s[40:43], s67 offen sc0 sc1\n\ts_add_u32 s67, s67, s90\n\t" \
  "buffer_load_dwordx4 v[148:151], %[trb], s[40:43], s67 offen sc0 sc1\n\ts_add_u32 s67, s67, s90\n\t" \
  "buffer_load_dwordx4 v[152:155], %[trb], s[40:43], s67 offen sc0 sc1\n\ts_add_u32 s67, s67, s90\n\t" \
  "buffer_load_dwordx4 v[156:159], %[trb], s[40:43], s67 offen sc0 sc1\n\ts_add_u32 s67, s67, s90\n\t" \
  "buffer_load_dwordx4 v[160:163], %[trb], s[40:43], s67 offen sc0 sc1\n\ts_add_u32 s67, s67, s90\n\t" \
  "buffer_load_dwordx4 v[164:167], %[trb], s[40:43], s67 offen sc0 sc1\n\ts_add_u32 s67, s67, s90\n\t" \
  "buffer_load_dwordx4 v[168:171], %[trb], s[40:43], s67 offen sc0 sc1\n\ts_add_u32 s67, s67, s90\n\t" \
  "buffer_load_dwordx4 v[172:175], %[trb], s[40:43], s67 offen sc0 sc1\n\ts_add_u32 s67, s67, s90\n\t" \
  "buffer_load_dwordx4 v[176:179], %[trb], s[40:43], s67 offen sc0 sc1\n\ts_add_u32 s67, s67, s90\n\t" \
  "buffer_load_dwordx4 v[180:183], %[trb], s[40:43], s67 offen sc0 sc1\n\ts_add_u32 s67, s67, s90\n\t" \
  "buffer_load_dwordx4 v[184:187], %[trb], s[40:43], s67 offen sc0 sc1\n\ts_add_u32 s67, s67, s90\n\t" \
  "buffer_load_dwordx4 v[188:191], %[trb], s[40:43], s67 offen sc0 sc1\n\ts_add_u32 s67, s67, s90\n\ts_waitcnt lgkmcnt(0)\n\t" \
  "Lloop_%=:\ns_waitcnt vmcnt(10)\n\t" \
  "v_mfma_f32_16x16x128_f8f6f4 v[208:211], a[0:7], v[144:151], 0\n\tbuffer_load_dwordx4 v[192:195], %[trb], s[40:43], s67 offen sc0 sc1\n\t" \
  "s_add_u32 s67, s67, s90\n\tds_write_b128 %[wr0], v[232:235] offset:384\n\t" \
  "v_mfma_f32_16x16x128_f8f6f4 v[212:215], a[8:15], v[144:151], 0\n\ts_waitcnt vmcnt(5)\n\tv_readlane_b32 s62, %[voff1], 0\n\t" \
  "v_readlane_b32 s63, %[voff1], 1\n\tv_readlane_b32 s64, %[voff1], 2\n\tv_readlane_b32 s65, %[voff1], 3\n\t" \
  "v_mfma_f32_16x16x128_f8f6f4 v[216:219], a[16:23], v[144:151], 0\n\tbuffer_load_dwordx4 v[196:199], %[trb], s[40:43], s67 offen sc0 sc1\n\t" \
  "s_add_u32 s67, s67, s90\n\tds_write_b128 %[wr1], v[236:239] offset:384\n\t" \
  "v_mfma_f32_16x16x128_f8f6f4 v[220:223], a[24:31], v[144:151], 0\n\ts_min_u32 s62, s62, s63\n\ts_min_u32 s64, s64, s65\n\ts_min_u32 s62, s62, s64\n\t" \
  "s_or_b32 s62, s62, s71\n\ts_cmp_ge_u32 s62, s37\n\ts_cselect_b32 s66, 1, 0\n\t" \
  "v_mfma_f32_16x16x128_f8f6f4 v[224:227], a[32:39], v[144:151], 0\n\tbuffer_load_dwordx4 v[200:203], %[trb], s[40:43], s67 offen sc0 sc1\n\t" \
  "s_add_u32 s67, s67, s90\n\t" \
  "v_mfma_f32_16x16x128_f8f6f4 v[228:231], a[40:47], v[144:151], 0\n\ts_waitcnt lgkmcnt(0)\n\ts_barrier\n\tds_read_b128 v[240:243], %[rd] offset:0\n\t" \
  "v_mfma_f32_16x16x128_f8f6f4 v[232:235], a[48:55], v[144:151], 0\n\tbuffer_load_dwordx4 v[204:207], %[trb], s[40:43], s67 offen sc0 sc1\n\t" \
  "s_add_u32 s67, s67, s90\n\tds_read_b128 v[244:247], %[rd] offset:8192\n\tds_read_b128 v[248:251], %[rd] offset:16384\n\t" \
  "v_mfma_f32_16x16x128_f8f6f4 v[236:239], a[56:63], v[144:151], 0\n\ts_cmp_lg_u32 s72, 0\n\ts_cselect_b64 s[40:41], s[28:29], s[98:99]\n\t" \
  "s_cselect_b32 s42, s95, s94\n\ts_cselect_b32 s90, 64, 0x400\n\tds_read_b128 v[252:255], %[rd] offset:24576\n\ts_waitcnt vmcnt(12)\n\t" \
  "v_mfma_f32_16x16x128_f8f6f4 v[208:211], a[64:71], v[152:159], v[208:211]\n\tv_bfi_b32 %[trb], s72, %[voffX], %[voffA]\n\ts_mov_b64 s[44:45], s[40:41]\n\t" \
  "s_lshl_b32 s67, s70, 16\n\ts_cmp_lg_u32 s66, 0\n\ts_cselect_b32 s46, s42, 0\n\t" \
  "v_mfma_f32_16x16x128_f8f6f4 v[212:215], a[72:79], v[152:159], v[212:215]\n\ts_waitcnt lgkmcnt(2)\n\tv_add_f32_e32 v240, v240, v244\n\t" \
  "v_add_f32_e32 v241, v241, v245\n\tv_add_f32_e32 v242, v242, v246\n\tv_add_f32_e32 v243, v243, v247\n\t" \
  "v_mfma_f32_16x16x128_f8f6f4 v[216:219], a[80:87], v[152:159], v[216:219]\n\tbuffer_load_dwordx4 v[144:147], %[trb], s[44:47], s67 offen sc0 sc1\n\t" \
  "s_add_u32 s67, s67, s90\n\tds_read_b128 v[244:247], %[rd] offset:256\n\ts_xor_b32 s86, s68, 0x100\n\t" \
  "v_mfma_f32_16x16x128_f8f6f4 v[220:223], a[88:95], v[152:159], v[220:223]\n\ts_waitcnt lgkmcnt(1)\n\tv_add_f32_e32 v248, v248, v252\n\t" \
  "v_add_f32_e32 v249, v249, v253\n\tv_add_f32_e32 v250, v250, v254\n\tv_add_f32_e32 v251, v251, v255\n\t" \
  "v_mfma_f32_16x16x128_f8f6f4 v[224:227], a[96:103], v[152:159], v[224:227]\n\tbuffer_load_dwordx4 v[148:151], %[trb], s[44:47], s67 offen sc0 sc1\n\t" \
  "s_add_u32 s67, s67, s90\n\tds_read_b128 v[252:255], %[rd] offset:8448\n\t" \
  "v_mfma_f32_16x16x128_f8f6f4 v[228:231], a[104:111], v[152:159], v[228:231]\n\tv_add_f32_e32 v240, v240, v248\n\tv_add_f32_e32 v241, v241, v249\n\t" \
  "v_add_f32_e32 v242, v242, v250\n\tv_add_f32_e32 v243, v243, v251\n\t" \
  "v_mfma_f32_16x16x128_f8f6f4 v[232:235], a[112:119], v[152:159], v[232:235]\n\tds_read_b128 v[248:251], %[rd] offset:16640\n\t" \
  "v_mfma_f32_16x16x128_f8f6f4 v[236:239], a[120:127], v[152:159], v[236:239]\n\tv_mul_f32_e32 v240, s39, v240\n\tv_mul_f32_e32 v241, s39, v241\n\t" \
  "v_mul_f32_e32 v242, s39, v242\n\tv_mul_f32_e32 v243, s39, v243\n\ts_waitcnt vmcnt(12)\n\t" \
  "v_mfma_f32_16x16x128_f8f6f4 v[208:211], a[128:135], v[160:167], v[208:211]\n\tbuffer_store_dwordx4 v[240:243], %[voffO], s[52:55], s68 offen sc1\n\t" \
  "ds_read_b128 v[240:243], %[rd] offset:24832\n\t" \
  "v_mfma_f32_16x16x128_f8f6f4 v[212:215], a[136:143], v[160:167], v[212:215]\n\ts_waitcnt lgkmcnt(2)\n\tv_add_f32_e32 v244, v244, v252\n\t" \
  "v_add_f32_e32 v245, v245, v253\n\tv_add_f32_e32 v246, v246, v254\n\tv_add_f32_e32 v247, v247, v255\n\t" \
  "v_mfma_f32_16x16x128_f8f6f4 v[216:219], a[144:151], v[160:167], v[216:219]\n\tbuffer_load_dwordx4 v[152:155], %[trb], s[44:47], s67 offen sc0 sc1\n\t" \
  "s_add_u32 s67, s67, s90\n\t" \
  "v_mfma_f32_16x16x128_f8f6f4 v[220:223], a[152:159], v[160:167], v[220:223]\n\ts_waitcnt lgkmcnt(1)\n\tv_add_f32_e32 v244, v244, v248\n\t" \
  "v_add_f32_e32 v245, v245, v249\n\tv_add_f32_e32 v246, v246, v250\n\tv_add_f32_e32 v247, v247, v251\n\t" \
  "v_mfma_f32_16x16x128_f8f6f4 v[224:227], a[160:167], v[160:167], v[224:227]\n\tbuffer_load_dwordx4 v[156:159], %[trb], s[44:47], s67 offen sc0 sc1\n\t" \
  "s_add_u32 s67, s67, s90\n\t" \
  "v_mfma_f32_16x16x128_f8f6f4 v[228:231], a[168:175], v[160:167], v[228:231]\n\ts_waitcnt lgkmcnt(0)\n\tv_add_f32_e32 v244, v244, v240\n\t" \
  "v_add_f32_e32 v245, v245, v241\n\tv_add_f32_e32 v246, v246, v242\n\tv_add_f32_e32 v247, v247, v243\n\t" \
  "v_mfma_f32_16x16x128_f8f6f4 v[232:235], a[176:183], v[160:167], v[232:235]\n\tv_mul_f32_e32 v244, s39, v244\n\tv_mul_f32_e32 v245, s39, v245\n\t" \
  "v_mul_f32_e32 v246, s39, v246\n\tv_mul_f32_e32 v247, s39, v247\n\tds_read_b128 v[248:251], %[voffA] offset:0\n\t" \
  "v_mfma_f32_16x16x128_f8f6f4 v[236:239], a[184:191], v[160:167], v[236:239]\n\tbuffer_store_dwordx4 v[244:247], %[voffO], s[52:55], s86 offen sc1\n\t" \
  "ds_read_b128 v[252:255], %[voffA] offset:8192\n\ts_waitcnt vmcnt(14)\n\t" \
  "v_mfma_f32_16x16x128_f8f6f4 v[208:211], a[192:199], v[168:175], v[208:211]\n\tbuffer_load_dwordx4 v[160:163], %[trb], s[44:47], s67 offen sc0 sc1\n\t" \
  "s_add_u32 s67, s67, s90\n\tds_read_b128 v[240:243], %[voffA] offset:1024\n\t" \
  "v_mfma_f32_16x16x128_f8f6f4 v[212:215], a[200:207], v[168:175], v[212:215]\n\ts_lshl_b32 s68, s69, 17\n\ts_or_b32 s68, s68, s93\n\t" \
  "s_add_u32 s77, s76, 2\n\tds_read_b128 v[244:247], %[voffA] offset:9216\n\t" \
  "v_mfma_f32_16x16x128_f8f6f4 v[216:219], a[208:215], v[168:175], v[216:219]\n\tbuffer_load_dwordx4 v[164:167], %[trb], s[44:47], s67 offen sc0 sc1\n\t" \
  "s_add_u32 s67, s67, s90\n\ts_waitcnt vmcnt(14) lgkmcnt(2)\n\t" \
  "v_mfma_f32_16x16x128_f8f6f4 v[208:211], v[248:255], v[176:183], v[208:211]\n\tds_read_b128 v[248:251], %[voffA] offset:2048\n\t" \
  "v_mfma_f32_16x16x128_f8f6f4 v[220:223], a[216:223], v[168:175], v[220:223]\n\ts_bitcmp1_b32 %[ownmask], s77\n\ts_cselect_b32 s75, -1, 0\n\t" \
  "s_cselect_b32 s73, s81, s80\n\ts_cselect_b32 s78, 0, 2\n\tds_read_b128 v[252:255], %[voffA] offset:10240\n\ts_waitcnt lgkmcnt(2)\n\t" \
  "v_mfma_f32_16x16x128_f8f6f4 v[212:215], v[240:247], v[176:183], v[212:215]\n\tds_read_b128 v[240:243], %[voffA] offset:3072\n\t" \
  "v_mfma_f32_16x16x128_f8f6f4 v[224:227], a[224:231], v[168:175], v[224:227]\n\ts_add_u32 s80, s80, s78\n\ts_sub_u32 s81, s81, s75\n\ts_mov_b32 s74, s75\n\t" \
  "ds_read_b128 v[244:247], %[voffA] offset:11264\n\t" \
  "v_mfma_f32_16x16x128_f8f6f4 v[228:231], a[232:239], v[168:175], v[228:231]\n\ts_sub_u32 s78, s77, s89\n\ts_add_u32 s79, s30, s78\n\t" \
  "s_cmp_ge_u32 s77, s89\n\ts_cselect_b32 s73, s79, s73\n\ts_waitcnt lgkmcnt(2)\n\t" \
  "v_mfma_f32_16x16x128_f8f6f4 v[216:219], v[248:255], v[176:183], v[216:219]\n\tds_read_b128 v[248:251], %[voffA] offset:4096\n\t" \
  "v_mfma_f32_16x16x128_f8f6f4 v[232:235], a[240:247], v[168:175], v[232:235]\n\ts_and_b32 s88, s70, 31\n\ts_cmp_lt_u32 s88, 28\n\t" \
  "s_cselect_b32 s88, 1, 0\n\ts_or_b32 s88, s88, s70\n\ts_lshl_b32 s88, s88, 10\n\tbuffer_load_dword %[voff0], %[voffF], s[48:51], s88 offen sc0 sc1\n\t" \
  "ds_read_b128 v[252:255], %[voffA] offset:12288\n\ts_waitcnt lgkmcnt(2)\n\t" \
  "v_mfma_f32_16x16x128_f8f6f4 v[220:223], v[240:247], v[176:183], v[220:223]\n\tds_read_b128 v[240:243], %[voffA] offset:5120\n\t" \
  "v_mfma_f32_16x16x128_f8f6f4 v[236:239], a[248:255], v[168:175], v[236:239]\n\tds_read_b128 v[244:247], %[voffA] offset:13312\n\ts_waitcnt vmcnt(13)\n\t" \
  "v_mfma_f32_16x16x128_f8f6f4 v[208:211], v[16:23], v[184:191], v[208:211]\n\tbuffer_load_dwordx4 v[168:171], %[trb], s[44:47], s67 offen sc0 sc1\n\t" \
  "s_add_u32 s67, s67, s90\n\ts_waitcnt lgkmcnt(2)\n\t" \
  "v_mfma_f32_16x16x128_f8f6f4 v[224:227], v[248:255], v[176:183], v[224:227]\n\tds_read_b128 v[248:251], %[voffA] offset:6144\n\t" \
  "v_mfma_f32_16x16x128_f8f6f4 v[212:215], v[24:31], v[184:191], v[212:215]\n\tbuffer_load_dwordx4 v[172:175], %[trb], s[44:47], s67 offen sc0 sc1\n\t" \
  "s_add_u32 s67, s67, s90\n\tds_read_b128 v[252:255], %[voffA] offset:14336\n\ts_waitcnt lgkmcnt(2)\n\t" \
  "v_mfma_f32_16x16x128_f8f6f4 v[228:231], v[240:247], v[176:183], v[228:231]\n\tds_read_b128 v[240:243], %[voffA] offset:7168\n\t" \
  "v_mfma_f32_16x16x128_f8f6f4 v[216:219], v[32:39], v[184:191], v[216:219]\n\ts_barrier\n\tds_read_b128 v[244:247], %[voffA] offset:15360\n\t" \
  "v_mfma_f32_16x16x128_f8f6f4 v[220:223], v[40:47], v[184:191], v[220:223]\n\ts_waitcnt lgkmcnt(2)\n\t" \
  "v_mfma_f32_16x16x128_f8f6f4 v[232:235], v[248:255], v[176:183], v[232:235]\n\tds_read_b128 v[248:251], %[voffA] offset:16384\n\t" \
  "v_mfma_f32_16x16x128_f8f6f4 v[224:227], v[48:55], v[184:191], v[224:227]\n\ts_and_b32 s88, s73, 31\n\ts_cmp_lt_u32 s88, 28\n\ts_cselect_b32 s88, 1, 0\n\t" \
  "s_or_b32 s88, s88, s73\n\ts_lshl_b32 s88, s88, 10\n\tbuffer_load_dword %[voff1], %[voffF], s[48:51], s88 offen sc0 sc1\n\t" \
  "ds_read_b128 v[252:255], %[voffA] offset:24576\n\ts_waitcnt lgkmcnt(2)\n\t" \
  "v_mfma_f32_16x16x128_f8f6f4 v[236:239], v[240:247], v[176:183], v[236:239]\n\tds_read_b128 v[240:243], %[voffA] offset:17408\n\t" \
  "v_mfma_f32_16x16x128_f8f6f4 v[228:231], v[56:63], v[184:191], v[228:231]\n\tbuffer_load_dwordx4 v[176:179], %[trb], s[44:47], s67 offen sc0 sc1\n\t" \
  "s_add_u32 s67, s67, s90\n\tds_read_b128 v[244:247], %[voffA] offset:25600\n\t" \
  "v_mfma_f32_16x16x128_f8f6f4 v[232:235], v[64:71], v[184:191], v[232:235]\n\tbuffer_load_dwordx4 v[180:183], %[trb], s[44:47], s67 offen sc0 sc1\n\t" \
  "s_add_u32 s67, s67, s90\n\ts_waitcnt vmcnt(14) lgkmcnt(2)\n\t" \
  "v_mfma_f32_16x16x128_f8f6f4 v[208:211], v[248:255], v[200:207], v[208:211]\n\tds_read_b128 v[248:251], %[voffA] offset:18432\n\t" \
  "v_mfma_f32_16x16x128_f8f6f4 v[236:239], v[72:79], v[184:191], v[236:239]\n\tbuffer_load_dwordx4 v[184:187], %[trb], s[44:47], s67 offen sc0 sc1\n\t" \
  "s_add_u32 s67, s67, s90\n\tds_read_b128 v[252:255], %[voffA] offset:26624\n\ts_waitcnt lgkmcnt(2)\n\t" \
  "v_mfma_f32_16x16x128_f8f6f4 v[212:215], v[240:247], v[200:207], v[212:215]\n\tds_read_b128 v[240:243], %[voffA] offset:19456\n\ts_waitcnt vmcnt(17)\n\t" \
  "v_mfma_f32_16x16x128_f8f6f4 v[208:211], v[80:87], v[192:199], v[208:211]\n\tbuffer_load_dwordx4 v[188:191], %[trb], s[44:47], s67 offen sc0 sc1\n\t" \
  "s_add_u32 s67, s67, s90\n\tds_read_b128 v[244:247], %[voffA] offset:27648\n\t" \
  "v_mfma_f32_16x16x128_f8f6f4 v[212:215], v[88:95], v[192:199], v[212:215]\n\ts_cmp_lg_u32 s66, 0\n\ts_cbranch_scc0 LlateM_%=\n\t" \
  "LlatebM_%=:\ns_waitcnt lgkmcnt(2)\n\t" \
  "v_mfma_f32_16x16x128_f8f6f4 v[216:219], v[248:255], v[200:207], v[216:219]\n\tds_read_b128 v[248:251], %[voffA] offset:20480\n\t" \
  "v_mfma_f32_16x16x128_f8f6f4 v[216:219], v[96:103], v[192:199], v[216:219]\n\tds_write_b128 %[wr0], v[208:211] offset:0\n\t" \
  "ds_read_b128 v[252:255], %[voffA] offset:28672\n\ts_waitcnt lgkmcnt(3)\n\t" \
  "v_mfma_f32_16x16x128_f8f6f4 v[220:223], v[240:247], v[200:207], v[220:223]\n\tds_write_b128 %[wr1], v[212:215] offset:0\n\t" \
  "ds_read_b128 v[240:243], %[voffA] offset:21504\n\t" \
  "v_mfma_f32_16x16x128_f8f6f4 v[220:223], v[104:111], v[192:199], v[220:223]\n\ts_add_u32 s76, s76, 1\n\ts_cmp_lt_u32 s76, s89\n\t" \
  "s_cselect_b64 vcc, -1, 0\n\tds_read_b128 v[244:247], %[voffA] offset:29696\n\t" \
  "v_mfma_f32_16x16x128_f8f6f4 v[224:227], v[112:119], v[192:199], v[224:227]\n\tds_write_b128 %[wr0], v[216:219] offset:128\n\ts_waitcnt lgkmcnt(4)\n\t" \
  "v_mfma_f32_16x16x128_f8f6f4 v[224:227], v[248:255], v[200:207], v[224:227]\n\tds_read_b128 v[248:251], %[voffA] offset:22528\n\t" \
  "v_mfma_f32_16x16x128_f8f6f4 v[228:231], v[120:127], v[192:199], v[228:231]\n\tds_write_b128 %[wr1], v[220:223] offset:128\n\t" \
  "ds_read_b128 v[252:255], %[voffA] offset:30720\n\ts_waitcnt lgkmcnt(4)\n\t" \
  "v_mfma_f32_16x16x128_f8f6f4 v[228:231], v[240:247], v[200:207], v[228:231]\n\ts_mov_b32 s69, s70\n\ts_mov_b32 s87, s71\n\t" \
  "ds_read_b128 v[240:243], %[voffA] offset:23552\n\t" \
  "v_mfma_f32_16x16x128_f8f6f4 v[232:235], v[128:135], v[192:199], v[232:235]\n\tds_write_b128 %[wr0], v[224:227] offset:256\n\t" \
  "ds_read_b128 v[244:247], %[voffA] offset:31744\n\ts_waitcnt lgkmcnt(3)\n\t" \
  "v_mfma_f32_16x16x128_f8f6f4 v[232:235], v[248:255], v[200:207], v[232:235]\n\ts_mov_b32 s70, s73\n\ts_mov_b32 s71, s74\n\ts_mov_b32 s72, s75\n\t" \
  "v_mfma_f32_16x16x128_f8f6f4 v[236:239], v[136:143], v[192:199], v[236:239]\n\tds_write_b128 %[wr1], v[228:231] offset:256\n\ts_waitcnt lgkmcnt(1)\n\t" \
  "v_mfma_f32_16x16x128_f8f6f4 v[236:239], v[240:247], v[200:207], v[236:239]\n\ts_cmp_lg_u32 s66, 0\n\ts_cbranch_scc0 LwaitB_%=\n\t" \
  "LbackB_%=:\ns_cbranch_vccnz Lloop_%=\n\ts_waitcnt vmcnt(10)\n\t" \
  "v_mfma_f32_16x16x128_f8f6f4 v[208:211], a[0:7], v[144:151], 0\n\tbuffer_load_dwordx4 v[192:195], %[trb], s[40:43], s67 offen sc0 sc1\n\t" \
  "s_add_u32 s67, s67, s90\n\tds_write_b128 %[wr0], v[232:235] offset:384\n\t" \
  "v_mfma_f32_16x16x128_f8f6f4 v[212:215], a[8:15], v[144:151], 0\n\tbuffer_load_dwordx4 v[196:199], %[trb], s[40:43], s67 offen sc0 sc1\n\t" \
  "s_add_u32 s67, s67, s90\n\ts_waitcnt vmcnt(6)\n\tv_readlane_b32 s62, %[voff1], 0\n\tv_readlane_b32 s63, %[voff1], 1\n\tv_readlane_b32 s64, %[voff1], 2\n\t" \
  "v_readlane_b32 s65, %[voff1], 3\n\t" \
  "v_mfma_f32_16x16x128_f8f6f4 v[216:219], a[16:23], v[144:151], 0\n\tbuffer_load_dwordx4 v[200:203], %[trb], s[40:43], s67 offen sc0 sc1\n\t" \
  "s_add_u32 s67, s67, s90\n\tds_write_b128 %[wr1], v[236:239] offset:384\n\t" \
  "v_mfma_f32_16x16x128_f8f6f4 v[220:223], a[24:31], v[144:151], 0\n\tbuffer_load_dwordx4 v[204:207], %[trb], s[40:43], s67 offen sc0 sc1\n\t" \
  "s_add_u32 s67, s67, s90\n\ts_min_u32 s62, s62, s63\n\ts_min_u32 s64, s64, s65\n\ts_min_u32 s62, s62, s64\n\ts_or_b32 s62, s62, s71\n\t" \
  "s_cmp_ge_u32 s62, s37\n\ts_cselect_b32 s66, 1, 0\n\ts_waitcnt vmcnt(12)\n\t" \
  "v_mfma_f32_16x16x128_f8f6f4 v[208:211], a[64:71], v[152:159], v[208:211]\n\ts_cmp_lg_u32 s66, 0\n\ts_cselect_b32 s46, s42, 0\n\t" \
  "s_andn2_b32 s46, s46, s71\n\ts_lshl_b32 s67, s70, 16\n\t" \
  "v_mfma_f32_16x16x128_f8f6f4 v[212:215], a[72:79], v[152:159], v[212:215]\n\ts_waitcnt lgkmcnt(0)\n\ts_barrier\n\t" \
  "ds_read_b128 v[240:243], %[rd] offset:0\n\t" \
  "v_mfma_f32_16x16x128_f8f6f4 v[216:219], a[80:87], v[152:159], v[216:219]\n\tbuffer_load_dwordx4 v[144:147], %[trb], s[44:47], s67 offen sc0 sc1\n\t" \
  "s_add_u32 s67, s67, s90\n\tds_read_b128 v[244:247], %[rd] offset:8192\n\tds_read_b128 v[248:251], %[rd] offset:16384\n\t" \
  "v_mfma_f32_16x16x128_f8f6f4 v[220:223], a[88:95], v[152:159], v[220:223]\n\tbuffer_load_dwordx4 v[148:151], %[trb], s[44:47], s67 offen sc0 sc1\n\t" \
  "s_add_u32 s67, s67, s90\n\tds_read_b128 v[252:255], %[rd] offset:24576\n\ts_waitcnt vmcnt(12)\n\t" \
  "v_mfma_f32_16x16x128_f8f6f4 v[208:211], a[128:135], v[160:167], v[208:211]\n\tbuffer_load_dwordx4 v[152:155], %[trb], s[44:47], s67 offen sc0 sc1\n\t" \
  "s_add_u32 s67, s67, s90\n\tds_read_b128 v[224:227], %[voffA] offset:0\n\t" \
  "v_mfma_f32_16x16x128_f8f6f4 v[212:215], a[136:143], v[160:167], v[212:215]\n\tds_read_b128 v[228:231], %[voffA] offset:8192\n\ts_and_b32 s88, s70, 31\n\t" \
  "s_cmp_lt_u32 s88, 28\n\ts_cselect_b32 s88, 1, 0\n\ts_or_b32 s88, s88, s70\n\ts_lshl_b32 s88, s88, 10\n\t" \
  "buffer_load_dword %[voff0], %[voffF], s[48:51], s88 offen sc0 sc1\n\ts_waitcnt lgkmcnt(4)\n\tv_add_f32_e32 v240, v240, v244\n\t" \
  "v_add_f32_e32 v241, v241, v245\n\tv_add_f32_e32 v242, v242, v246\n\tv_add_f32_e32 v243, v243, v247\n\t" \
  "v_mfma_f32_16x16x128_f8f6f4 v[216:219], a[144:151], v[160:167], v[216:219]\n\tbuffer_load_dwordx4 v[156:159], %[trb], s[44:47], s67 offen sc0 sc1\n\t" \
  "s_add_u32 s67, s67, s90\n\tds_read_b128 v[232:235], %[voffA] offset:1024\n\tds_read_b128 v[244:247], %[rd] offset:256\n\ts_xor_b32 s86, s68, 0x100\n\t" \
  "v_mfma_f32_16x16x128_f8f6f4 v[220:223], a[152:159], v[160:167], v[220:223]\n\tds_read_b128 v[236:239], %[voffA] offset:9216\n\ts_waitcnt lgkmcnt(5)\n\t" \
  "v_add_f32_e32 v248, v248, v252\n\tv_add_f32_e32 v249, v249, v253\n\tv_add_f32_e32 v250, v250, v254\n\tv_add_f32_e32 v251, v251, v255\n\t" \
  "s_waitcnt vmcnt(13)\n\t" \
  "v_mfma_f32_16x16x128_f8f6f4 v[208:211], a[192:199], v[168:175], v[208:211]\n\tbuffer_load_dwordx4 v[160:163], %[trb], s[44:47], s67 offen sc0 sc1\n\t" \
  "s_add_u32 s67, s67, s90\n\tds_read_b128 v[252:255], %[rd] offset:8448\n\ts_add_u32 s73, s70, 1\n\ts_and_b32 s78, s73, 31\n\ts_cselect_b32 s74, 0, -1\n\t" \
  "s_andn2_b32 s73, s73, s74\n\t" \
  "v_mfma_f32_16x16x128_f8f6f4 v[212:215], a[200:207], v[168:175], v[212:215]\n\tv_add_f32_e32 v240, v240, v248\n\tv_add_f32_e32 v241, v241, v249\n\t" \
  "v_add_f32_e32 v242, v242, v250\n\tv_add_f32_e32 v243, v243, v251\n\ts_waitcnt vmcnt(12) lgkmcnt(4)\n\t" \
  "v_mfma_f32_16x16x128_f8f6f4 v[208:211], v[224:231], v[176:183], v[208:211]\n\tbuffer_load_dwordx4 v[164:167], %[trb], s[44:47], s67 offen sc0 sc1\n\t" \
  "s_add_u32 s67, s67, s90\n\tds_read_b128 v[224:227], %[voffA] offset:2048\n\tds_read_b128 v[248:251], %[rd] offset:16640\n\ts_and_b32 s88, s73, 31\n\t" \
  "s_cmp_lt_u32 s88, 28\n\ts_cselect_b32 s88, 1, 0\n\ts_or_b32 s88, s88, s73\n\ts_lshl_b32 s88, s88, 10\n\t" \
  "buffer_load_dword %[voff1], %[voffF], s[48:51], s88 offen sc0 sc1\n\t" \
  "v_mfma_f32_16x16x128_f8f6f4 v[216:219], a[208:215], v[168:175], v[216:219]\n\tds_read_b128 v[228:231], %[voffA] offset:10240\n\t" \
  "v_mul_f32_e32 v240, s39, v240\n\tv_mul_f32_e32 v241, s39, v241\n\tv_mul_f32_e32 v242, s39, v242\n\tv_mul_f32_e32 v243, s39, v243\n\t" \
  "s_waitcnt lgkmcnt(4)\n\t" \
  "v_mfma_f32_16x16x128_f8f6f4 v[212:215], v[232:239], v[176:183], v[212:215]\n\tds_read_b128 v[232:235], %[voffA] offset:3072\n\t" \
  "buffer_store_dwordx4 v[240:243], %[voffO], s[52:55], s68 offen sc1\n\tds_read_b128 v[240:243], %[rd] offset:24832\n\t" \
  "v_mfma_f32_16x16x128_f8f6f4 v[220:223], a[216:223], v[168:175], v[220:223]\n\tds_read_b128 v[236:239], %[voffA] offset:11264\n\ts_waitcnt lgkmcnt(6)\n\t" \
  "v_add_f32_e32 v244, v244, v252\n\tv_add_f32_e32 v245, v245, v253\n\tv_add_f32_e32 v246, v246, v254\n\tv_add_f32_e32 v247, v247, v255\n\t" \
  "s_waitcnt lgkmcnt(3)\n\t" \
  "v_mfma_f32_16x16x128_f8f6f4 v[216:219], v[224:231], v[176:183], v[216:219]\n\tbuffer_load_dwordx4 v[168:171], %[trb], s[44:47], s67 offen sc0 sc1\n\t" \
  "s_add_u32 s67, s67, s90\n\tds_read_b128 v[224:227], %[voffA] offset:16384\n\ts_waitcnt vmcnt(14)\n\t" \
  "v_mfma_f32_16x16x128_f8f6f4 v[208:211], v[16:23], v[184:191], v[208:211]\n\tbuffer_load_dwordx4 v[172:175], %[trb], s[44:47], s67 offen sc0 sc1\n\t" \
  "s_add_u32 s67, s67, s90\n\tds_read_b128 v[228:231], %[voffA] offset:24576\n\ts_waitcnt lgkmcnt(6)\n\tv_add_f32_e32 v244, v244, v248\n\t" \
  "v_add_f32_e32 v245, v245, v249\n\tv_add_f32_e32 v246, v246, v250\n\tv_add_f32_e32 v247, v247, v251\n\ts_waitcnt lgkmcnt(2)\n\t" \
  "v_mfma_f32_16x16x128_f8f6f4 v[220:223], v[232:239], v[176:183], v[220:223]\n\tds_read_b128 v[232:235], %[voffA] offset:17408\n\t" \
  "v_mfma_f32_16x16x128_f8f6f4 v[212:215], v[24:31], v[184:191], v[212:215]\n\tbuffer_load_dwordx4 v[176:179], %[trb], s[44:47], s67 offen sc0 sc1\n\t" \
  "s_add_u32 s67, s67, s90\n\tds_read_b128 v[236:239], %[voffA] offset:25600\n\ts_waitcnt lgkmcnt(5)\n\tv_add_f32_e32 v244, v244, v240\n\t" \
  "v_add_f32_e32 v245, v245, v241\n\tv_add_f32_e32 v246, v246, v242\n\tv_add_f32_e32 v247, v247, v243\n\t" \
  "v_mfma_f32_16x16x128_f8f6f4 v[216:219], v[32:39], v[184:191], v[216:219]\n\tbuffer_load_dwordx4 v[180:183], %[trb], s[44:47], s67 offen sc0 sc1\n\t" \
  "s_add_u32 s67, s67, s90\n\tv_mul_f32_e32 v244, s39, v244\n\tv_mul_f32_e32 v245, s39, v245\n\tv_mul_f32_e32 v246, s39, v246\n\t" \
  "v_mul_f32_e32 v247, s39, v247\n\t" \
  "v_mfma_f32_16x16x128_f8f6f4 v[220:223], v[40:47], v[184:191], v[220:223]\n\tbuffer_store_dwordx4 v[244:247], %[voffO], s[52:55], s86 offen sc1\n\t" \
  "s_waitcnt vmcnt(16)\n\t" \
  "v_mfma_f32_16x16x128_f8f6f4 v[208:211], v[80:87], v[192:199], v[208:211]\n\tbuffer_load_dwordx4 v[184:187], %[trb], s[44:47], s67 offen sc0 sc1\n\t" \
  "s_add_u32 s67, s67, s90\n\ts_barrier\n\ts_lshl_b32 s68, s69, 17\n\ts_or_b32 s68, s68, s93\n\ts_waitcnt vmcnt(15) lgkmcnt(2)\n\t" \
  "v_mfma_f32_16x16x128_f8f6f4 v[208:211], v[224:231], v[200:207], v[208:211]\n\tbuffer_load_dwordx4 v[188:191], %[trb], s[44:47], s67 offen sc0 sc1\n\t" \
  "s_add_u32 s67, s67, s90\n\tds_read_b128 v[224:227], %[voffA] offset:18432\n\t" \
  "v_mfma_f32_16x16x128_f8f6f4 v[212:215], v[88:95], v[192:199], v[212:215]\n\tds_read_b128 v[228:231], %[voffA] offset:26624\n\ts_cmp_lg_u32 s66, 0\n\t" \
  "s_cbranch_scc0 LlateF_%=\n\t" \
  "LlatebF_%=:\ns_waitcnt lgkmcnt(2)\n\t" \
  "v_mfma_f32_16x16x128_f8f6f4 v[212:215], v[232:239], v[200:207], v[212:215]\n\tds_read_b128 v[232:235], %[voffA] offset:19456\n\ts_cmp_eq_u32 s71, 0\n\t" \
  "s_cselect_b64 vcc, -1, 0\n\t" \
  "v_mfma_f32_16x16x128_f8f6f4 v[216:219], v[96:103], v[192:199], v[216:219]\n\tds_read_b128 v[236:239], %[voffA] offset:27648\n\t" \
  "ds_write_b128 %[wr0], v[208:211] offset:0\n\ts_mov_b32 s69, s70\n\ts_mov_b32 s87, s71\n\ts_waitcnt lgkmcnt(3)\n\t" \
  "v_mfma_f32_16x16x128_f8f6f4 v[216:219], v[224:231], v[200:207], v[216:219]\n\t" \
  "v_mfma_f32_16x16x128_f8f6f4 v[220:223], v[104:111], v[192:199], v[220:223]\n\tds_write_b128 %[wr1], v[212:215] offset:0\n\ts_mov_b32 s70, s73\n\t" \
  "s_mov_b32 s71, s74\n\ts_waitcnt lgkmcnt(2)\n\t" \
  "v_mfma_f32_16x16x128_f8f6f4 v[220:223], v[232:239], v[200:207], v[220:223]\n\ts_cmp_lg_u32 s66, 0\n\ts_cbranch_scc0 LwaitC_%=\n\t" \
  "LbackC_%=:\n" \
  "Lhalf_%=:\ns_waitcnt vmcnt(10)\n\t" \
  "v_mfma_f32_16x16x128_f8f6f4 v[208:211], a[0:7], v[144:151], 0\n\tbuffer_load_dwordx4 v[192:195], %[trb], s[40:43], s67 offen sc0 sc1\n\t" \
  "s_add_u32 s67, s67, s90\n\tds_write_b128 %[wr0], v[216:219] offset:128\n\t" \
  "v_mfma_f32_16x16x128_f8f6f4 v[212:215], a[8:15], v[144:151], 0\n\tbuffer_load_dwordx4 v[196:199], %[trb], s[40:43], s67 offen sc0 sc1\n\t" \
  "s_add_u32 s67, s67, s90\n\ts_waitcnt vmcnt(10)\n\tv_readlane_b32 s62, %[voff1], 0\n\tv_readlane_b32 s63, %[voff1], 1\n\t" \
  "v_readlane_b32 s64, %[voff1], 2\n\tv_readlane_b32 s65, %[voff1], 3\n\t" \
  "v_mfma_f32_16x16x128_f8f6f4 v[216:219], a[16:23], v[144:151], 0\n\tbuffer_load_dwordx4 v[200:203], %[trb], s[40:43], s67 offen sc0 sc1\n\t" \
  "s_add_u32 s67, s67, s90\n\tds_write_b128 %[wr1], v[220:223] offset:128\n\t" \
  "v_mfma_f32_16x16x128_f8f6f4 v[220:223], a[24:31], v[144:151], 0\n\tbuffer_load_dwordx4 v[204:207], %[trb], s[40:43], s67 offen sc0 sc1\n\t" \
  "s_add_u32 s67, s67, s90\n\ts_min_u32 s62, s62, s63\n\ts_min_u32 s64, s64, s65\n\ts_min_u32 s62, s62, s64\n\ts_or_b32 s62, s62, s71\n\t" \
  "s_cmp_ge_u32 s62, s37\n\ts_cselect_b32 s66, 1, 0\n\ts_waitcnt vmcnt(12)\n\t" \
  "v_mfma_f32_16x16x128_f8f6f4 v[208:211], a[64:71], v[152:159], v[208:211]\n\ts_cmp_lg_u32 s66, 0\n\ts_cselect_b32 s46, s42, 0\n\t" \
  "s_andn2_b32 s46, s46, s71\n\ts_lshl_b32 s67, s70, 16\n\t" \
  "v_mfma_f32_16x16x128_f8f6f4 v[212:215], a[72:79], v[152:159], v[212:215]\n\ts_waitcnt lgkmcnt(0)\n\ts_barrier\n\t" \
  "ds_read_b128 v[240:243], %[rd] offset:0\n\tds_read_b128 v[244:247], %[rd] offset:8192\n\t" \
  "v_mfma_f32_16x16x128_f8f6f4 v[216:219], a[80:87], v[152:159], v[216:219]\n\tbuffer_load_dwordx4 v[144:147], %[trb], s[44:47], s67 offen sc0 sc1\n\t" \
  "s_add_u32 s67, s67, s90\n\tds_read_b128 v[248:251], %[rd] offset:16384\n\tds_read_b128 v[252:255], %[rd] offset:24576\n\t" \
  "v_mfma_f32_16x16x128_f8f6f4 v[220:223], a[88:95], v[152:159], v[220:223]\n\tbuffer_load_dwordx4 v[148:151], %[trb], s[44:47], s67 offen sc0 sc1\n\t" \
  "s_add_u32 s67, s67, s90\n\ts_waitcnt vmcnt(12)\n\t" \
  "v_mfma_f32_16x16x128_f8f6f4 v[208:211], a[128:135], v[160:167], v[208:211]\n\tbuffer_load_dwordx4 v[152:155], %[trb], s[44:47], s67 offen sc0 sc1\n\t" \
  "s_add_u32 s67, s67, s90\n\tds_read_b128 v[224:227], %[voffA] offset:0\n\t" \
  "v_mfma_f32_16x16x128_f8f6f4 v[212:215], a[136:143], v[160:167], v[212:215]\n\tds_read_b128 v[228:231], %[voffA] offset:8192\n\ts_and_b32 s88, s70, 31\n\t" \
  "s_cmp_lt_u32 s88, 28\n\ts_cselect_b32 s88, 1, 0\n\ts_or_b32 s88, s88, s70\n\ts_lshl_b32 s88, s88, 10\n\t" \
  "buffer_load_dword %[voff0], %[voffF], s[48:51], s88 offen sc0 sc1\n\t" \
  "v_mfma_f32_16x16x128_f8f6f4 v[216:219], a[144:151], v[160:167], v[216:219]\n\tbuffer_load_dwordx4 v[156:159], %[trb], s[44:47], s67 offen sc0 sc1\n\t" \
  "s_add_u32 s67, s67, s90\n\tds_read_b128 v[232:235], %[voffA] offset:1024\n\t" \
  "v_mfma_f32_16x16x128_f8f6f4 v[220:223], a[152:159], v[160:167], v[220:223]\n\tds_read_b128 v[236:239], %[voffA] offset:9216\n\ts_waitcnt lgkmcnt(6)\n\t" \
  "v_add_f32_e32 v240, v240, v244\n\tv_add_f32_e32 v241, v241, v245\n\tv_add_f32_e32 v242, v242, v246\n\tv_add_f32_e32 v243, v243, v247\n\t" \
  "s_waitcnt vmcnt(13)\n\t" \
  "v_mfma_f32_16x16x128_f8f6f4 v[208:211], a[192:199], v[168:175], v[208:211]\n\tbuffer_load_dwordx4 v[160:163], %[trb], s[44:47], s67 offen sc0 sc1\n\t" \
  "s_add_u32 s67, s67, s90\n\ts_add_u32 s73, s70, 1\n\ts_and_b32 s78, s73, 31\n\ts_cselect_b32 s74, 0, -1\n\ts_andn2_b32 s73, s73, s74\n\t" \
  "v_mfma_f32_16x16x128_f8f6f4 v[212:215], a[200:207], v[168:175], v[212:215]\n\ts_waitcnt lgkmcnt(4)\n\tv_add_f32_e32 v248, v248, v252\n\t" \
  "v_add_f32_e32 v249, v249, v253\n\tv_add_f32_e32 v250, v250, v254\n\tv_add_f32_e32 v251, v251, v255\n\ts_and_b32 s88, s73, 31\n\ts_cmp_lt_u32 s88, 28\n\t" \
  "s_cselect_b32 s88, 1, 0\n\ts_or_b32 s88, s88, s73\n\ts_lshl_b32 s88, s88, 10\n\tbuffer_load_dword %[voff1], %[voffF], s[48:51], s88 offen sc0 sc1\n\t" \
  "s_waitcnt vmcnt(13) lgkmcnt(2)\n\t" \
  "v_mfma_f32_16x16x128_f8f6f4 v[208:211], v[224:231], v[176:183], v[208:211]\n\tbuffer_load_dwordx4 v[164:167], %[trb], s[44:47], s67 offen sc0 sc1\n\t" \
  "s_add_u32 s67, s67, s90\n\tds_read_b128 v[224:227], %[voffA] offset:2048\n\t" \
  "v_mfma_f32_16x16x128_f8f6f4 v[216:219], a[208:215], v[168:175], v[216:219]\n\tds_read_b128 v[228:231], %[voffA] offset:10240\n\t" \
  "v_add_f32_e32 v240, v240, v248\n\tv_add_f32_e32 v241, v241, v249\n\tv_add_f32_e32 v242, v242, v250\n\tv_add_f32_e32 v243, v243, v251\n\t" \
  "s_waitcnt lgkmcnt(2)\n\t" \
  "v_mfma_f32_16x16x128_f8f6f4 v[212:215], v[232:239], v[176:183], v[212:215]\n\tds_read_b128 v[232:235], %[voffA] offset:3072\n\t" \
  "v_mfma_f32_16x16x128_f8f6f4 v[220:223], a[216:223], v[168:175], v[220:223]\n\tds_read_b128 v[236:239], %[voffA] offset:11264\n\t" \
  "v_mul_f32_e32 v240, s39, v240\n\tv_mul_f32_e32 v241, s39, v241\n\tv_mul_f32_e32 v242, s39, v242\n\tv_mul_f32_e32 v243, s39, v243\n\t" \
  "s_waitcnt lgkmcnt(2)\n\t" \
  "v_mfma_f32_16x16x128_f8f6f4 v[216:219], v[224:231], v[176:183], v[216:219]\n\tbuffer_load_dwordx4 v[168:171], %[trb], s[44:47], s67 offen sc0 sc1\n\t" \
  "s_add_u32 s67, s67, s90\n\tds_read_b128 v[224:227], %[voffA] offset:16384\n\ts_waitcnt vmcnt(13)\n\t" \
  "v_mfma_f32_16x16x128_f8f6f4 v[208:211], v[16:23], v[184:191], v[208:211]\n\tbuffer_load_dwordx4 v[172:175], %[trb], s[44:47], s67 offen sc0 sc1\n\t" \
  "s_add_u32 s67, s67, s90\n\tds_read_b128 v[228:231], %[voffA] offset:24576\n\tbuffer_store_dwordx4 v[240:243], %[voffO], s[52:55], s68 offen sc1\n\t" \
  "s_waitcnt lgkmcnt(2)\n\t" \
  "v_mfma_f32_16x16x128_f8f6f4 v[220:223], v[232:239], v[176:183], v[220:223]\n\tds_read_b128 v[232:235], %[voffA] offset:17408\n\t" \
  "v_mfma_f32_16x16x128_f8f6f4 v[212:215], v[24:31], v[184:191], v[212:215]\n\tbuffer_load_dwordx4 v[176:179], %[trb], s[44:47], s67 offen sc0 sc1\n\t" \
  "s_add_u32 s67, s67, s90\n\tds_read_b128 v[236:239], %[voffA] offset:25600\n\t" \
  "v_mfma_f32_16x16x128_f8f6f4 v[216:219], v[32:39], v[184:191], v[216:219]\n\tbuffer_load_dwordx4 v[180:183], %[trb], s[44:47], s67 offen sc0 sc1\n\t" \
  "s_add_u32 s67, s67, s90\n\t" \
  "v_mfma_f32_16x16x128_f8f6f4 v[220:223], v[40:47], v[184:191], v[220:223]\n\ts_waitcnt vmcnt(15)\n\t" \
  "v_mfma_f32_16x16x128_f8f6f4 v[208:211], v[80:87], v[192:199], v[208:211]\n\tbuffer_load_dwordx4 v[184:187], %[trb], s[44:47], s67 offen sc0 sc1\n\t" \
  "s_add_u32 s67, s67, s90\n\ts_barrier\n\ts_lshl_b32 s68, s69, 17\n\ts_or_b32 s68, s68, s93\n\ts_waitcnt vmcnt(14) lgkmcnt(2)\n\t" \
  "v_mfma_f32_16x16x128_f8f6f4 v[208:211], v[224:231], v[200:207], v[208:211]\n\tbuffer_load_dwordx4 v[188:191], %[trb], s[44:47], s67 offen sc0 sc1\n\t" \
  "s_add_u32 s67, s67, s90\n\tds_read_b128 v[224:227], %[voffA] offset:18432\n\t" \
  "v_mfma_f32_16x16x128_f8f6f4 v[212:215], v[88:95], v[192:199], v[212:215]\n\tds_read_b128 v[228:231], %[voffA] offset:26624\n\ts_cmp_lg_u32 s66, 0\n\t" \
  "s_cbranch_scc0 LlateH_%=\n\t" \
  "LlatebH_%=:\ns_waitcnt lgkmcnt(2)\n\t" \
  "v_mfma_f32_16x16x128_f8f6f4 v[212:215], v[232:239], v[200:207], v[212:215]\n\tds_read_b128 v[232:235], %[voffA] offset:19456\n\ts_cmp_eq_u32 s71, 0\n\t" \
  "s_cselect_b64 vcc, -1, 0\n\t" \
  "v_mfma_f32_16x16x128_f8f6f4 v[216:219], v[96:103], v[192:199], v[216:219]\n\tds_read_b128 v[236:239], %[voffA] offset:27648\n\t" \
  "ds_write_b128 %[wr0], v[208:211] offset:0\n\ts_mov_b32 s69, s70\n\ts_mov_b32 s87, s71\n\ts_waitcnt lgkmcnt(3)\n\t" \
  "v_mfma_f32_16x16x128_f8f6f4 v[216:219], v[224:231], v[200:207], v[216:219]\n\t" \
  "v_mfma_f32_16x16x128_f8f6f4 v[220:223], v[104:111], v[192:199], v[220:223]\n\tds_write_b128 %[wr1], v[212:215] offset:0\n\ts_mov_b32 s70, s73\n\t" \
  "s_mov_b32 s71, s74\n\ts_waitcnt lgkmcnt(2)\n\t" \
  "v_mfma_f32_16x16x128_f8f6f4 v[220:223], v[232:239], v[200:207], v[220:223]\n\ts_cmp_lg_u32 s66, 0\n\ts_cbranch_scc0 LwaitD_%=\n\t" \
  "LbackD_%=:\ns_cbranch_vccnz Lhalf_%=\n\ts_nop 9\n\tds_write_b128 %[wr0], v[216:219] offset:128\n\tds_write_b128 %[wr1], v[220:223] offset:128\n\t" \
  "s_waitcnt lgkmcnt(0)\n\ts_barrier\n\tds_read_b128 v[240:243], %[rd] offset:0\n\tds_read_b128 v[244:247], %[rd] offset:8192\n\t" \
  "ds_read_b128 v[248:251], %[rd] offset:16384\n\tds_read_b128 v[252:255], %[rd] offset:24576\n\ts_waitcnt lgkmcnt(2)\n\tv_add_f32_e32 v240, v240, v244\n\t" \
  "v_add_f32_e32 v241, v241, v245\n\tv_add_f32_e32 v242, v242, v246\n\tv_add_f32_e32 v243, v243, v247\n\ts_waitcnt lgkmcnt(0)\n\t" \
  "v_add_f32_e32 v248, v248, v252\n\tv_add_f32_e32 v249, v249, v253\n\tv_add_f32_e32 v250, v250, v254\n\tv_add_f32_e32 v251, v251, v255\n\t" \
  "v_add_f32_e32 v240, v240, v248\n\tv_add_f32_e32 v241, v241, v249\n\tv_add_f32_e32 v242, v242, v250\n\tv_add_f32_e32 v243, v243, v251\n\t" \
  "v_mul_f32_e32 v240, s39, v240\n\tv_mul_f32_e32 v241, s39, v241\n\tv_mul_f32_e32 v242, s39, v242\n\tv_mul_f32_e32 v243, s39, v243\n\t" \
  "buffer_store_dwordx4 v[240:243], %[voffO], s[52:55], s68 offen sc1\n\ts_branch Lend_%=\n\t" \
  "LwaitB_%=:\ns_cmp_lg_u32 s38, 0\n\ts_cbranch_scc1 LgoB_%=\n\ts_memrealtime s[58:59]\n\ts_mov_b32 s77, 0\n\t" \
  "LspinB_%=:\ns_and_b32 s88, s69, 31\n\ts_cmp_lt_u32 s88, 28\n\ts_cselect_b32 s88, 1, 0\n\ts_or_b32 s88, s88, s69\n\ts_lshl_b32 s88, s88, 10\n\t" \
  "buffer_load_dword %[voff0], %[voffF], s[48:51], s88 offen sc0 sc1\n\ts_waitcnt vmcnt(0)\n\tv_readlane_b32 s62, %[voff0], 0\n\t" \
  "v_readlane_b32 s63, %[voff0], 1\n\tv_readlane_b32 s64, %[voff0], 2\n\tv_readlane_b32 s65, %[voff0], 3\n\ts_min_u32 s62, s62, s63\n\t" \
  "s_min_u32 s64, s64, s65\n\ts_min_u32 s62, s62, s64\n\ts_or_b32 s62, s62, s87\n\ts_cmp_ge_u32 s62, s37\n\ts_cbranch_scc1 LgoB_%=\n\ts_sleep 2\n\t" \
  "s_add_u32 s77, s77, 1\n\ts_and_b32 s78, s77, 63\n\ts_cbranch_scc1 LspinB_%=\n\tv_mov_b32 %[pvoff], 0\n\t" \
  "global_load_dword %[voff0], %[pvoff], s[82:83] sc1\n\ts_memrealtime s[60:61]\n\ts_waitcnt vmcnt(0) lgkmcnt(0)\n\tv_readfirstlane_b32 s78, %[voff0]\n\t" \
  "s_cmp_eq_u32 s78, s37\n\ts_cbranch_scc1 LtmoB_%=\n\ts_sub_u32 s60, s60, s58\n\ts_subb_u32 s61, s61, s59\n\ts_cmp_lg_u32 s61, 0\n\t" \
  "s_cbranch_scc1 LtmoB_%=\n\ts_cmp_le_u32 s60, 50000000\n\ts_cbranch_scc1 LspinB_%=\n\t" \
  "LtmoB_%=:\nv_mov_b32 %[voff0], s37\n\tglobal_store_dword %[pvoff], %[voff0], s[82:83] sc1\n\ts_mov_b32 s38, 1\n\tv_mov_b32 v16, 0x7f7f7f7f\n\t" \
  "v_mov_b32 v24, 0x7f7f7f7f\n\tv_mov_b32 v32, 0x7f7f7f7f\n\tv_mov_b32 v40, 0x7f7f7f7f\n\tv_mov_b32 v48, 0x7f7f7f7f\n\tv_mov_b32 v56, 0x7f7f7f7f\n\t" \
  "v_mov_b32 v64, 0x7f7f7f7f\n\tv_mov_b32 v72, 0x7f7f7f7f\n\ts_waitcnt vmcnt(0)\n\t" \
  "LgoB_%=:\ns_lshl_b32 s67, s69, 16\n\tbuffer_load_dwordx4 v[144:147], %[trb], s[40:43], s67 offen sc0 sc1\n\ts_add_u32 s67, s67, s90\n\t" \
  "buffer_load_dwordx4 v[148:151], %[trb], s[40:43], s67 offen sc0 sc1\n\ts_add_u32 s67, s67, s90\n\t" \
  "buffer_load_dwordx4 v[152:155], %[trb], s[40:43], s67 offen sc0 sc1\n\ts_add_u32 s67, s67, s90\n\t" \
  "buffer_load_dwordx4 v[156:159], %[trb], s[40:43], s67 offen sc0 sc1\n\ts_add_u32 s67, s67, s90\n\t" \
  "buffer_load_dwordx4 v[160:163], %[trb], s[40:43], s67 offen sc0 sc1\n\ts_add_u32 s67, s67, s90\n\t" \
  "buffer_load_dwordx4 v[164:167], %[trb], s[40:43], s67 offen sc0 sc1\n\ts_add_u32 s67, s67, s90\n\t" \
  "buffer_load_dwordx4 v[168:171], %[trb], s[40:43], s67 offen sc0 sc1\n\ts_add_u32 s67, s67, s90\n\t" \
  "buffer_load_dwordx4 v[172:175], %[trb], s[40:43], s67 offen sc0 sc1\n\ts_add_u32 s67, s67, s90\n\t" \
  "buffer_load_dwordx4 v[176:179], %[trb], s[40:43], s67 offen sc0 sc1\n\ts_add_u32 s67, s67, s90\n\t" \
  "buffer_load_dwordx4 v[180:183], %[trb], s[40:43], s67 offen sc0 sc1\n\ts_add_u32 s67, s67, s90\n\t" \
  "buffer_load_dwordx4 v[184:187], %[trb], s[40:43], s67 offen sc0 sc1\n\ts_add_u32 s67, s67, s90\n\t" \
  "buffer_load_dwordx4 v[188:191], %[trb], s[40:43], s67 offen sc0 sc1\n\ts_add_u32 s67, s67, s90\n\ts_waitcnt lgkmcnt(0)\n\ts_branch LbackB_%=\n\t" \
  "LwaitC_%=:\ns_cmp_lg_u32 s38, 0\n\ts_cbranch_scc1 LgoC_%=\n\ts_memrealtime s[58:59]\n\ts_mov_b32 s77, 0\n\t" \
  "LspinC_%=:\ns_and_b32 s88, s69, 31\n\ts_cmp_lt_u32 s88, 28\n\ts_cselect_b32 s88, 1, 0\n\ts_or_b32 s88, s88, s69\n\ts_lshl_b32 s88, s88, 10\n\t" \
  "buffer_load_dword %[voff0], %[voffF], s[48:51], s88 offen sc0 sc1\n\ts_waitcnt vmcnt(0)\n\tv_readlane_b32 s62, %[voff0], 0\n\t" \
  "v_readlane_b32 s63, %[voff0], 1\n\tv_readlane_b32 s64, %[voff0], 2\n\tv_readlane_b32 s65, %[voff0], 3\n\ts_min_u32 s62, s62, s63\n\t" \
  "s_min_u32 s64, s64, s65\n\ts_min_u32 s62, s62, s64\n\ts_or_b32 s62, s62, s87\n\ts_cmp_ge_u32 s62, s37\n\ts_cbranch_scc1 LgoC_%=\n\ts_sleep 2\n\t" \
  "s_add_u32 s77, s77, 1\n\ts_and_b32 s78, s77, 63\n\ts_cbranch_scc1 LspinC_%=\n\tv_mov_b32 %[pvoff], 0\n\t" \
  "global_load_dword %[voff0], %[pvoff], s[82:83] sc1\n\ts_memrealtime s[60:61]\n\ts_waitcnt vmcnt(0) lgkmcnt(0)\n\tv_readfirstlane_b32 s78, %[voff0]\n\t" \
  "s_cmp_eq_u32 s78, s37\n\ts_cbranch_scc1 LtmoC_%=\n\ts_sub_u32 s60, s60, s58\n\ts_subb_u32 s61, s61, s59\n\ts_cmp_lg_u32 s61, 0\n\t" \
  "s_cbranch_scc1 LtmoC_%=\n\ts_cmp_le_u32 s60, 50000000\n\ts_cbranch_scc1 LspinC_%=\n\t" \
  "LtmoC_%=:\nv_mov_b32 %[voff0], s37\n\tglobal_store_dword %[pvoff], %[voff0], s[82:83] sc1\n\ts_mov_b32 s38, 1\n\tv_mov_b32 v16, 0x7f7f7f7f\n\t" \
  "v_mov_b32 v24, 0x7f7f7f7f\n\tv_mov_b32 v32, 0x7f7f7f7f\n\tv_mov_b32 v40, 0x7f7f7f7f\n\tv_mov_b32 v48, 0x7f7f7f7f\n\tv_mov_b32 v56, 0x7f7f7f7f\n\t" \
  "v_mov_b32 v64, 0x7f7f7f7f\n\tv_mov_b32 v72, 0x7f7f7f7f\n\ts_waitcnt vmcnt(0)\n\t" \
  "LgoC_%=:\ns_lshl_b32 s67, s69, 16\n\tbuffer_load_dwordx4 v[144:147], %[trb], s[40:43], s67 offen sc0 sc1\n\ts_add_u32 s67, s67, s90\n\t" \
  "buffer_load_dwordx4 v[148:151], %[trb], s[40:43], s67 offen sc0 sc1\n\ts_add_u32 s67, s67, s90\n\t" \
  "buffer_load_dwordx4 v[152:155], %[trb], s[40:43], s67 offen sc0 sc1\n\ts_add_u32 s67, s67, s90\n\t" \
  "buffer_load_dwordx4 v[156:159], %[trb], s[40:43], s67 offen sc0 sc1\n\ts_add_u32 s67, s67, s90\n\t" \
  "buffer_load_dwordx4 v[160:163], %[trb], s[40:43], s67 offen sc0 sc1\n\ts_add_u32 s67, s67, s90\n\t" \
  "buffer_load_dwordx4 v[164:167], %[trb], s[40:43], s67 offen sc0 sc1\n\ts_add_u32 s67, s67, s90\n\t" \
  "buffer_load_dwordx4 v[168:171], %[trb], s[40:43], s67 offen sc0 sc1\n\ts_add_u32 s67, s67, s90\n\t" \
  "buffer_load_dwordx4 v[172:175], %[trb], s[40:43], s67 offen sc0 sc1\n\ts_add_u32 s67, s67, s90\n\t" \
  "buffer_load_dwordx4 v[176:179], %[trb], s[40:43], s67 offen sc0 sc1\n\ts_add_u32 s67, s67, s90\n\t" \
  "buffer_load_dwordx4 v[180:183], %[trb], s[40:43], s67 offen sc0 sc1\n\ts_add_u32 s67, s67, s90\n\t" \
  "buffer_load_dwordx4 v[184:187], %[trb], s[40:43], s67 offen sc0 sc1\n\ts_add_u32 s67, s67, s90\n\t" \
  "buffer_load_dwordx4 v[188:191], %[trb], s[40:43], s67 offen sc0 sc1\n\ts_add_u32 s67, s67, s90\n\ts_waitcnt lgkmcnt(0)\n\ts_branch LbackC_%=\n\t" \
  "LwaitD_%=:\ns_cmp_lg_u32 s38, 0\n\ts_cbranch_scc1 LgoD_%=\n\ts_memrealtime s[58:59]\n\ts_mov_b32 s77, 0\n\t" \
  "LspinD_%=:\ns_and_b32 s88, s69, 31\n\ts_cmp_lt_u32 s88, 28\n\ts_cselect_b32 s88, 1, 0\n\ts_or_b32 s88, s88, s69\n\ts_lshl_b32 s88, s88, 10\n\t" \
  "buffer_load_dword %[voff0], %[voffF], s[48:51], s88 offen sc0 sc1\n\ts_waitcnt vmcnt(0)\n\tv_readlane_b32 s62, %[voff0], 0\n\t" \
  "v_readlane_b32 s63, %[voff0], 1\n\tv_readlane_b32 s64, %[voff0], 2\n\tv_readlane_b32 s65, %[voff0], 3\n\ts_min_u32 s62, s62, s63\n\t" \
  "s_min_u32 s64, s64, s65\n\ts_min_u32 s62, s62, s64\n\ts_or_b32 s62, s62, s87\n\ts_cmp_ge_u32 s62, s37\n\ts_cbranch_scc1 LgoD_%=\n\ts_sleep 2\n\t" \
  "s_add_u32 s77, s77, 1\n\ts_and_b32 s78, s77, 63\n\ts_cbranch_scc1 LspinD_%=\n\tv_mov_b32 %[pvoff], 0\n\t" \
  "global_load_dword %[voff0], %[pvoff], s[82:83] sc1\n\ts_memrealtime s[60:61]\n\ts_waitcnt vmcnt(0) lgkmcnt(0)\n\tv_readfirstlane_b32 s78, %[voff0]\n\t" \
  "s_cmp_eq_u32 s78, s37\n\ts_cbranch_scc1 LtmoD_%=\n\ts_sub_u32 s60, s60, s58\n\ts_subb_u32 s61, s61, s59\n\ts_cmp_lg_u32 s61, 0\n\t" \
  "s_cbranch_scc1 LtmoD_%=\n\ts_cmp_le_u32 s60, 50000000\n\ts_cbranch_scc1 LspinD_%=\n\t" \
  "LtmoD_%=:\nv_mov_b32 %[voff0], s37\n\tglobal_store_dword %[pvoff], %[voff0], s[82:83] sc1\n\ts_mov_b32 s38, 1\n\tv_mov_b32 v16, 0x7f7f7f7f\n\t" \
  "v_mov_b32 v24, 0x7f7f7f7f\n\tv_mov_b32 v32, 0x7f7f7f7f\n\tv_mov_b32 v40, 0x7f7f7f7f\n\tv_mov_b32 v48, 0x7f7f7f7f\n\tv_mov_b32 v56, 0x7f7f7f7f\n\t" \
  "v_mov_b32 v64, 0x7f7f7f7f\n\tv_mov_b32 v72, 0x7f7f7f7f\n\ts_waitcnt vmcnt(0)\n\t" \
  "LgoD_%=:\ns_lshl_b32 s67, s69, 16\n\tbuffer_load_dwordx4 v[144:147], %[trb], s[40:43], s67 offen sc0 sc1\n\ts_add_u32 s67, s67, s90\n\t" \
  "buffer_load_dwordx4 v[148:151], %[trb], s[40:43], s67 offen sc0 sc1\n\ts_add_u32 s67, s67, s90\n\t" \
  "buffer_load_dwordx4 v[152:155], %[trb], s[40:43], s67 offen sc0 sc1\n\ts_add_u32 s67, s67, s90\n\t" \
  "buffer_load_dwordx4 v[156:159], %[trb], s[40:43], s67 offen sc0 sc1\n\ts_add_u32 s67, s67, s90\n\t" \
  "buffer_load_dwordx4 v[160:163], %[trb], s[40:43], s67 offen sc0 sc1\n\ts_add_u32 s67, s67, s90\n\t" \
  "buffer_load_dwordx4 v[164:167], %[trb], s[40:43], s67 offen sc0 sc1\n\ts_add_u32 s67, s67, s90\n\t" \
  "buffer_load_dwordx4 v[168:171], %[trb], s[40:43], s67 offen sc0 sc1\n\ts_add_u32 s67, s67, s90\n\t" \
  "buffer_load_dwordx4 v[172:175], %[trb], s[40:43], s67 offen sc0 sc1\n\ts_add_u32 s67, s67, s90\n\t" \
  "buffer_load_dwordx4 v[176:179], %[trb], s[40:43], s67 offen sc0 sc1\n\ts_add_u32 s67, s67, s90\n\t" \
  "buffer_load_dwordx4 v[180:183], %[trb], s[40:43], s67 offen sc0 sc1\n\ts_add_u32 s67, s67, s90\n\t" \
  "buffer_load_dwordx4 v[184:187], %[trb], s[40:43], s67 offen sc0 sc1\n\ts_add_u32 s67, s67, s90\n\t" \
  "buffer_load_dwordx4 v[188:191], %[trb], s[40:43], s67 offen sc0 sc1\n\ts_add_u32 s67, s67, s90\n\ts_waitcnt lgkmcnt(0)\n\ts_branch LbackD_%=\n\t" \
  "LlateM_%=:\ns_waitcnt vmcnt(7)\n\tv_readlane_b32 s62, %[voff0], 0\n\tv_readlane_b32 s63, %[voff0], 1\n\tv_readlane_b32 s64, %[voff0], 2\n\t" \
  "v_readlane_b32 s65, %[voff0], 3\n\ts_min_u32 s62, s62, s63\n\ts_min_u32 s64, s64, s65\n\ts_min_u32 s62, s62, s64\n\ts_or_b32 s62, s62, s71\n\t" \
  "s_cmp_ge_u32 s62, s37\n\ts_cbranch_scc0 LlatebM_%=\n\ts_mov_b32 s66, 1\n\ts_lshl_b32 s67, s70, 16\n\t" \
  "buffer_load_dwordx4 v[144:147], %[trb], s[40:43], s67 offen sc0 sc1\n\ts_add_u32 s67, s67, s90\n\t" \
  "buffer_load_dwordx4 v[148:151], %[trb], s[40:43], s67 offen sc0 sc1\n\ts_add_u32 s67, s67, s90\n\t" \
  "buffer_load_dwordx4 v[152:155], %[trb], s[40:43], s67 offen sc0 sc1\n\ts_add_u32 s67, s67, s90\n\t" \
  "buffer_load_dwordx4 v[156:159], %[trb], s[40:43], s67 offen sc0 sc1\n\ts_add_u32 s67, s67, s90\n\t" \
  "buffer_load_dwordx4 v[160:163], %[trb], s[40:43], s67 offen sc0 sc1\n\ts_add_u32 s67, s67, s90\n\t" \
  "buffer_load_dwordx4 v[164:167], %[trb], s[40:43], s67 offen sc0 sc1\n\ts_add_u32 s67, s67, s90\n\t" \
  "buffer_load_dwordx4 v[168:171], %[trb], s[40:43], s67 offen sc0 sc1\n\ts_add_u32 s67, s67, s90\n\t" \
  "buffer_load_dwordx4 v[172:175], %[trb], s[40:43], s67 offen sc0 sc1\n\ts_add_u32 s67, s67, s90\n\t" \
  "buffer_load_dwordx4 v[176:179], %[trb], s[40:43], s67 offen sc0 sc1\n\ts_add_u32 s67, s67, s90\n\t" \
  "buffer_load_dwordx4 v[180:183], %[trb], s[40:43], s67 offen sc0 sc1\n\ts_add_u32 s67, s67, s90\n\t" \
  "buffer_load_dwordx4 v[184:187], %[trb], s[40:43], s67 offen sc0 sc1\n\ts_add_u32 s67, s67, s90\n\t" \
  "buffer_load_dwordx4 v[188:191], %[trb], s[40:43], s67 offen sc0 sc1\n\ts_add_u32 s67, s67, s90\n\ts_branch LlatebM_%=\n\t" \
  "LlateF_%=:\ns_waitcnt vmcnt(12)\n\tv_readlane_b32 s62, %[voff0], 0\n\tv_readlane_b32 s63, %[voff0], 1\n\tv_readlane_b32 s64, %[voff0], 2\n\t" \
  "v_readlane_b32 s65, %[voff0], 3\n\ts_min_u32 s62, s62, s63\n\ts_min_u32 s64, s64, s65\n\ts_min_u32 s62, s62, s64\n\ts_or_b32 s62, s62, s71\n\t" \
  "s_cmp_ge_u32 s62, s37\n\ts_cbranch_scc0 LlatebF_%=\n\ts_mov_b32 s66, 1\n\ts_lshl_b32 s67, s70, 16\n\t" \
  "buffer_load_dwordx4 v[144:147], %[trb], s[40:43], s67 offen sc0 sc1\n\ts_add_u32 s67, s67, s90\n\t" \
  "buffer_load_dwordx4 v[148:151], %[trb], s[40:43], s67 offen sc0 sc1\n\ts_add_u32 s67, s67, s90\n\t" \
  "buffer_load_dwordx4 v[152:155], %[trb], s[40:43], s67 offen sc0 sc1\n\ts_add_u32 s67, s67, s90\n\t" \
  "buffer_load_dwordx4 v[156:159], %[trb], s[40:43], s67 offen sc0 sc1\n\ts_add_u32 s67, s67, s90\n\t" \
  "buffer_load_dwordx4 v[160:163], %[trb], s[40:43], s67 offen sc0 sc1\n\ts_add_u32 s67, s67, s90\n\t" \
  "buffer_load_dwordx4 v[164:167], %[trb], s[40:43], s67 offen sc0 sc1\n\ts_add_u32 s67, s67, s90\n\t" \
  "buffer_load_dwordx4 v[168:171], %[trb], s[40:43], s67 offen sc0 sc1\n\ts_add_u32 s67, s67, s90\n\t" \
  "buffer_load_dwordx4 v[172:175], %[trb], s[40:43], s67 offen sc0 sc1\n\ts_add_u32 s67, s67, s90\n\t" \
  "buffer_load_dwordx4 v[176:179], %[trb], s[40:43], s67 offen sc0 sc1\n\ts_add_u32 s67, s67, s90\n\t" \
  "buffer_load_dwordx4 v[180:183], %[trb], s[40:43], s67 offen sc0 sc1\n\ts_add_u32 s67, s67, s90\n\t" \
  "buffer_load_dwordx4 v[184:187], %[trb], s[40:43], s67 offen sc0 sc1\n\ts_add_u32 s67, s67, s90\n\t" \
  "buffer_load_dwordx4 v[188:191], %[trb], s[40:43], s67 offen sc0 sc1\n\ts_add_u32 s67, s67, s90\n\ts_branch LlatebF_%=\n\t" \
  "LlateH_%=:\ns_waitcnt vmcnt(11)\n\tv_readlane_b32 s62, %[voff0], 0\n\tv_readlane_b32 s63, %[voff0], 1\n\tv_readlane_b32 s64, %[voff0], 2\n\t" \
  "v_readlane_b32 s65, %[voff0], 3\n\ts_min_u32 s62, s62, s63\n\ts_min_u32 s64, s64, s65\n\ts_min_u32 s62, s62, s64\n\ts_or_b32 s62, s62, s71\n\t" \
  "s_cmp_ge_u32 s62, s37\n\ts_cbranch_scc0 LlatebH_%=\n\ts_mov_b32 s66, 1\n\ts_lshl_b32 s67, s70, 16\n\t" \
  "buffer_load_dwordx4 v[144:147], %[trb], s[40:43], s67 offen sc0 sc1\n\ts_add_u32 s67, s67, s90\n\t" \
  "buffer_load_dwordx4 v[148:151], %[trb], s[40:43], s67 offen sc0 sc1\n\ts_add_u32 s67, s67, s90\n\t" \
  "buffer_load_dwordx4 v[152:155], %[trb], s[40:43], s67 offen sc0 sc1\n\ts_add_u32 s67, s67, s90\n\t" \
  "buffer_load_dwordx4 v[156:159], %[trb], s[40:43], s67 offen sc0 sc1\n\ts_add_u32 s67, s67, s90\n\t" \
  "buffer_load_dwordx4 v[160:163], %[trb], s[40:43], s67 offen sc0 sc1\n\ts_add_u32 s67, s67, s90\n\t" \
  "buffer_load_dwordx4 v[164:167], %[trb], s[40:43], s67 offen sc0 sc1\n\ts_add_u32 s67, s67, s90\n\t" \
  "buffer_load_dwordx4 v[168:171], %[trb], s[40:43], s67 offen sc0 sc1\n\ts_add_u32 s67, s67, s90\n\t" \
  "buffer_load_dwordx4 v[172:175], %[trb], s[40:43], s67 offen sc0 sc1\n\ts_add_u32 s67, s67, s90\n\t" \
  "buffer_load_dwordx4 v[176:179], %[trb], s[40:43], s67 offen sc0 sc1\n\ts_add_u32 s67, s67, s90\n\t" \
  "buffer_load_dwordx4 v[180:183], %[trb], s[40:43], s67 offen sc0 sc1\n\ts_add_u32 s67, s67, s90\n\t" \
  "buffer_load_dwordx4 v[184:187], %[trb], s[40:43], s67 offen sc0 sc1\n\ts_add_u32 s67, s67, s90\n\t" \
  "buffer_load_dwordx4 v[188:191], %[trb], s[40:43], s67 offen sc0 sc1\n\ts_add_u32 s67, s67, s90\n\ts_branch LlatebH_%=\n\t" \
  "Lend_%=:\ns_mov_b32 m0, s36\n\t"
#define GEMM_KH 4
#define GEMM_FSH 10
#define GEMM_CLOBBERS \
  "v16", "v17", "v18", "v19", "v20", "v21", "v22", "v23", "v24", "v25", "v26", "v27", "v28", "v29", "v30", "v31", "v32", "v33", "v34", "v35", "v36", "v37", "v38", "v39", \
  "v40", "v41", "v42", "v43", "v44", "v45", "v46", "v47", "v48", "v49", "v50", "v51", "v52", "v53", "v54", "v55", "v56", "v57", "v58", "v59", "v60", "v61", "v62", "v63", \
  "v64", "v65", "v66", "v67", "v68", "v69", "v70", "v71", "v72", "v73", "v74", "v75", "v76", "v77", "v78", "v79", "v80", "v81", "v82", "v83", "v84", "v85", "v86", "v87", \
  "v88", "v89", "v90", "v91", "v92", "v93", "v94", "v95", "v96", "v97", "v98", "v99", "v100", "v101", "v102", "v103", "v104", "v105", "v106", "v107", "v108", "v109", "v110", "v111", \
  "v112", "v113", "v114", "v115", "v116", "v117", "v118", "v119", "v120", "v121", "v122", "v123", "v124", "v125", "v126", "v127", "v128", "v129", "v130", "v131", "v132", "v133", "v134", "v135", \
  "v136", "v137", "v138", "v139", "v140", "v141", "v142", "v143", "v144", "v145", "v146", "v147", "v148", "v149", "v150", "v151", "v152", "v153", "v154", "v155", "v156", "v157", "v158", "v159", \
  "v160", "v161", "v162", "v163", "v164", "v165", "v166", "v167", "v168", "v169", "v170", "v171", "v172", "v173", "v174", "v175", "v176", "v177", "v178", "v179", "v180", "v181", "v182", "v183", \
  "v184", "v185", "v186", "v187", "v188", "v189", "v190", "v191", "v192", "v193", "v194", "v195", "v196", "v197", "v198", "v199", "v200", "v201", "v202", "v203", "v204", "v205", "v206", "v207", \
  "v208", "v209", "v210", "v211", "v212", "v213", "v214", "v215", "v216", "v217", "v218", "v219", "v220", "v221", "v222", "v223", "v224", "v225", "v226", "v227", "v228", "v229", "v230", "v231", \
  "v232", "v233", "v234", "v235", "v236", "v237", "v238", "v239", "v240", "v241", "v242", "v243", "v244", "v245", "v246", "v247", "v248", "v249", "v250", "v251", "v252", "v253", "v254", "v255", \
  "a0", "a1", "a2", "a3", "a4", "a5", "a6", "a7", "a8", "a9", "a10", "a11", "a12", "a13", "a14", "a15", "a16", "a17", "a18", "a19", "a20", "a21", "a22", "a23", \
  "a24", "a25", "a26", "a27", "a28", "a29", "a30", "a31", "a32", "a33", "a34", "a35", "a36", "a37", "a38", "a39", "a40", "a41", "a42", "a43", "a44", "a45", "a46", "a47", \
  "a48", "a49", "a50", "a51", "a52", "a53", "a54", "a55", "a56", "a57", "a58", "a59", "a60", "a61", "a62", "a63", "a64", "a65", "a66", "a67", "a68", "a69", "a70", "a71", \
  "a72", "a73", "a74", "a75", "a76", "a77", "a78", "a79", "a80", "a81", "a82", "a83", "a84", "a85", "a86", "a87", "a88", "a89", "a90", "a91", "a92", "a93", "a94", "a95", \
  "a96", "a97", "a98", "a99", "a100", "a101", "a102", "a103", "a104", "a105", "a106", "a107", "a108", "a109", "a110", "a111", "a112", "a113", "a114", "a115", "a116", "a117", "a118", "a119", \
  "a120", "a121", "a122", "a123", "a124", "a125", "a126", "a127", "a128", "a129", "a130", "a131", "a132", "a133", "a134", "a135", "a136", "a137", "a138", "a139", "a140", "a141", "a142", "a143", \
  "a144", "a145", "a146", "a147", "a148", "a149", "a150", "a151", "a152", "a153", "a154", "a155", "a156", "a157", "a158", "a159", "a160", "a161", "a162", "a163", "a164", "a165", "a166", "a167", \
  "a168", "a169", "a170", "a171", "a172", "a173", "a174", "a175", "a176", "a177", "a178", "a179", "a180", "a181", "a182", "a183", "a184", "a185", "a186", "a187", "a188", "a189", "a190", "a191", \
  "a192", "a193", "a194", "a195", "a196", "a197", "a198", "a199", "a200", "a201", "a202", "a203", "a204", "a205", "a206", "a207", "a208", "a209", "a210", "a211", "a212", "a213", "a214", "a215", \
  "a216", "a217", "a218", "a219", "a220", "a221", "a222", "a223", "a224", "a225", "a226", "a227", "a228", "a229", "a230", "a231", "a232", "a233", "a234", "a235", "a236", "a237", "a238", "a239", \
  "a240", "a241", "a242", "a243", "a244", "a245", "a246", "a247", "a248", "a249", "a250", "a251", "a252", "a253", "a254", "a255", "s28", "s29", "s30", "s36", "s37", "s38", "s39", "s40", \
  "s41", "s42", "s43", "s44", "s45", "s46", "s47", "s48", "s49", "s50", "s51", "s52", "s53", "s54", "s55", "s56", "s57", "s58", "s59", "s60", "s61", "s62", "s63", "s64", \
  "s65", "s66", "s67", "s68", "s69", "s70", "s71", "s72", "s73", "s74", "s75", "s76", "s77", "s78", "s79", "s80", "s81", "s82", "s83", "s84", "s85", "s86", "s87", "s88", \
  "s89", "s90", "s91", "s92", "s93", "s94", "s95", "s96", "s97", "s98", "s99"

__global__ __launch_bounds__(256, 1) void gemm_ag_kernel(Params p) {
  __shared__ __attribute__((aligned(16))) unsigned char smem[LDS_BYTES];
  const int lane = threadIdx.x & 63;
  const int wid = __builtin_amdgcn_readfirstlane(threadIdx.x >> 6);
  const int bid = blockIdx.x;
  unsigned rank = (unsigned)p.rank;
  unsigned char* myws = p.myws;
  unsigned xk = __builtin_amdgcn_s_getreg((31 << 11) | 20) & 7u;
  unsigned char* dws = p.peer[xk];
  const unsigned char* px = p.x;
  asm volatile("" : "+s"(rank), "+s"(myws), "+s"(dws), "+s"(px));
  if (rank >= (unsigned)NR) return;

  if (__builtin_expect(bid < NHELP, 1)) {
    const unsigned slice = (unsigned)((bid >> 3) * 4 + wid);
    unsigned long long* tw = (unsigned long long*)(myws + (TICKET_OFF + slice * 128u));
    unsigned left = 0u;
    for (bool again = false;; again = true) {
      const unsigned long long add = again ? 0ull : 1ull << 32 | 1u << (4u * xk);
      unsigned long long old;
      if (xk != 0u) old = send_stream(px, dws, rank, slice, tw, add, again, lane);
      else {
        add_one_lane(old, tw, add);
        asm volatile("s_waitcnt vmcnt(0)" : "+v"(old) :: "memory");
        old = first_lane(old);
      }
      if (!again && ((unsigned)(old >> 32) & 7u) == 7u) {
        const unsigned got = (unsigned)old + (unsigned)add;
        for (unsigned k = 1; k < NR; ++k) left |= ((got >> (4u * k)) & 15u) == 0u ? 1u << k : 0u;
        add_one_lane(old, tw, 0ull - got);
        asm volatile("s_waitcnt vmcnt(0)" :: "v"(old) : "memory");
      }
      if (left == 0u) break;
      xk = (unsigned)__builtin_ctz(left) & 7u; left &= left - 1u;
      dws = p.peer[xk];
    }
    return;
  }

  const int xcd = bid & 7, jx = (bid - NHELP) >> 3;
  constexpr int XG = 16 / XCOLS;
  const int tt = (xcd / XG) * PER_XCD + jx;
  const int cg = (xcd % XG) * XCOLS + tt % XCOLS, g = tt / XCOLS;
  if ((unsigned)(size_t)(LAS unsigned char*)smem != 0u) return;
  asm volatile("" :: "s"((LAS unsigned char*)smem) : "memory");
  const int q = wid, g4 = lane >> 4, i16 = lane & 15;
  unsigned* ctr = (unsigned*)(myws + (CTR_OFF + (unsigned)(bid * 4 + wid) * 128u));
  const unsigned par = g & 1, nown = g < 4 ? 3 : 2, ownb = rank * NBLK + (unsigned)(2 * g + (g < 4 ? g : 4));
  const unsigned ownmask = nown == 3 ? (unsigned)OWNPOS : ((unsigned)OWNPOS & ~(0x80000000u >> __builtin_clz((unsigned)OWNPOS)));
  const unsigned pk = (((rank + 1u + (unsigned)(g >> 1)) & 7u) * NBLK + par) | (16u - GEMM_KH / 2 + nown) << 8 | (unsigned)q << 16 | ownb << 20;
  const unsigned voffA = q * STAGE_PER_WAVE + lane * 16;
  const unsigned voffX = i16 * KDIM + g4 * 16;
  const unsigned voffO = (4 * q + g4) * (NLOC * 4) + cg * 512 + i16 * 16, voffF = (lane & 3) * FU;
  static_assert(1u << GEMM_FSH == FB, "flags");
  const unsigned wr0 = RED_OFF + q * 8192 + i16 * 512 + ((g4 ^ (i16 & 7)) * 16), wr1 = wr0 ^ 64u;
  const unsigned rd = RED_OFF + (4 * q + g4) * 512 + ((i16 ^ ((4 * q + g4) & 7)) * 16);
  unsigned trb = q * STAGE_PER_WAVE + (16 * g4 + (i16 >> 1)) * 128 + (i16 & 1) * 8 + ((((i16 >> 2) & 3) | (((g4 ^ par) & 1) << 2)) * 16);
  const unsigned sw0 = (unsigned)((lane & 7) ^ ((lane >> 4) & 3));
  unsigned voff0 = (lane >> 3) * NGLOB + sw0 * 16, voff1 = (lane >> 3) * NGLOB + (sw0 ^ 4u) * 16;
  constexpr int RPI = 64 / XCOLS, PFROWS = 4096;
  static_assert(3 * 4 * PER_XCD * RPI >= PFROWS, "prefetch");
  unsigned pvoff = (lane / XCOLS) * NGLOB + (lane % XCOLS) * 128;
  const unsigned char* pf[3];
#pragma unroll
  for (int i = 0; i < 3; ++i) {
    int u = (i * 4 * PER_XCD + jx * 4 + q) * RPI; u = u < PFROWS ? u : PFROWS - RPI;
    const int h = u >> 8, kb = (0x74653210 >> (4 * (h >> 1))) & 7;
    pf[i] = p.w + rank * NLOC + (xcd % XG) * (XCOLS * 128) + (size_t)(((u >> 6) & 3) * 1024 + kb * 128 + (h & 1) * 64 + (u & 63)) * NGLOB;
  }
  asm volatile(GEMM_ASM
               : [voff0] "+v"(voff0), [voff1] "+v"(voff1), [pvoff] "+v"(pvoff), [trb] "+v"(trb)
               : [voffA] "v"(voffA), [voffO] "v"(voffO), [voffF] "v"(voffF), [wr0] "v"(wr0), [wr1] "v"(wr1), [rd] "v"(rd),
                 [voffX] "v"(voffX), [ctr] "s"(ctr), [w] "s"(p.w + (size_t)q * 1024 * NGLOB + rank * NLOC + cg * 128),
                 [pf0] "s"(pf[0]), [pf1] "s"(pf[1]), [pf2] "s"(pf[2]), [sx] "s"(p.sx), [sw] "s"(p.sw), [myws] "s"(myws), [out] "s"(p.out),
                 [pk] "s"(pk), [ownmask] "s"(ownmask), [xb] "s"(p.x - (size_t)rank * SHARD + q * 1024), [xsz] "s"((rank + 1u) * SHARD - q * 1024)
               : "memory", "scc", "vcc", GEMM_CLOBBERS);
}

unsigned near_peers(int rank) {
  const unsigned grp = 0xb2u, usual = (((grp >> rank) & 1u) ? grp : ~grp & 0xffu) & ~(1u << rank);
  int dev = -1, count = 0;
  if (hipGetDevice(&dev) != hipSuccess || hipGetDeviceCount(&count) != hipSuccess || dev != rank || count != NR) return usual;
  unsigned node[NR], near_nodes[3], n = 0;
  for (int d = 0; d < NR; ++d) {
    char bus[32], path[96];
    if (hipDeviceGetPCIBusId(bus, sizeof bus, d) != hipSuccess) return usual;
    for (char* c = bus; *c; ++c) if (*c >= 'A' && *c <= 'Z') *c += 'a' - 'A';
    snprintf(path, sizeof path, "/sys/bus/pci/devices/%s/xgmi_port_num", bus);
    FILE* f = fopen(path, "r");
    if (!f) return usual;
    unsigned a, pa, b, pb, lines = 0;
    while (fscanf(f, " %x:%x -> %x:%x", &a, &pa, &b, &pb) == 4) {
      node[d] = a; ++lines;
      if (d == rank && pa >= 1 && pa <= 3 && n < 3) near_nodes[n++] = b;
    }
    fclose(f);
    if (lines != NR - 1) return usual;
  }
  unsigned m = 0;
  for (int d = 0; d < NR; ++d)
    for (unsigned i = 0; i < n; ++i) if (d != rank && node[d] == near_nodes[i]) m |= 1u << d;
  return __builtin_popcount(m) == 3 ? m : usual;
}

}

extern "C" void kernel_launch(int rank, int nranks, ncclComm_t comm,
                              hipStream_t stream,
                              void* const* d_in, const int* in_sizes, int n_in,
                              void* d_out, int out_size,
                              const int* in_dtypes, int out_dtype,
                              void* const* d_out_all,
                              void* const* d_ws_all, size_t ws_bytes) {
  (void)comm; (void)in_dtypes; (void)out_dtype; (void)d_out_all;
  if (nranks != NR || (unsigned)rank >= (unsigned)NR || n_in < 4 || ws_bytes < WS_NEED ||
      in_sizes[0] != (int)SHARD || in_sizes[1] != KDIM * NGLOB || out_size != NR * MLOC * NLOC) {
    fprintf(stderr, "kernel_launch: unexpected problem shape\n");
    return;
  }
  Params p;
  p.x = (const unsigned char*)d_in[0];
  p.w = (const unsigned char*)d_in[1];
  p.sx = (const float*)d_in[2];
  p.sw = (const float*)d_in[3];
  p.out = (float*)d_out;
  const unsigned nearm = near_peers(rank);
  int near = 1, far = 4;
  p.myws = p.peer[0] = (unsigned char*)d_ws_all[rank];
  for (int r = 0; r < NR; ++r)
    if (r != rank) p.peer[((nearm >> r) & 1u) ? near++ : far++] = (unsigned char*)d_ws_all[r];
  p.rank = rank;
  p.pad = 0;
  hipLaunchKernelGGL(gemm_ag_kernel, dim3(NWG), dim3(256), 0, stream, p);
  HIP_CHECK(hipGetLastError());
}
```
